```python
import numpy as np

kernel_name = "gpu_gemm_a2a_m4096_k4096_n4096_bf16_none"


ABSMAX_RTOL = 0.008

COMPUTE_DTYPE = "bfloat16"
EPILOGUE = "none"
IN_QUANT = None
OUT_QUANT = None
IN_DTYPES = {"x": "bfloat16", "w_mat": "bfloat16"}
WORLD_SIZE = 8

def _to_bf16(x):
    u = np.ascontiguousarray(x, dtype=np.float32).view(np.uint32)
    return ((u + 0x7FFF + ((u >> 16) & 1)) & 0xFFFF0000).view(np.float32)

def setup_inputs():

    x = _to_bf16(np.random.randn(4096, 4096).astype(np.float32))
    w_mat = _to_bf16((np.random.randn(4096, 4096) / np.sqrt(4096)).astype(np.float32))
    return {"x": x, "w_mat": w_mat}

def _epilogue(y):
    return y

def reference(x, w_mat):
    y = (x @ w_mat).astype(np.float32)
    return _epilogue(y)
```

```cpp
#include <hip/hip_runtime.h>
#include <rccl/rccl.h>
#include <cstdio>
#include <cstdint>

#define HIP_CHECK(expr) do { hipError_t _e = (expr); if (_e != hipSuccess) { \
    fprintf(stderr, "HIP error %s at %s:%d\n", hipGetErrorString(_e), __FILE__, __LINE__); \
    return; } } while (0)

namespace {
typedef short bf16x8 __attribute__((ext_vector_type(8)));
typedef short bf16x4 __attribute__((ext_vector_type(4)));
typedef float f32x4 __attribute__((ext_vector_type(4)));
typedef unsigned u32x4 __attribute__((ext_vector_type(4)));
typedef unsigned u32x2 __attribute__((ext_vector_type(2)));
#define LAS __attribute__((address_space(3)))

constexpr int NR = 8, MS = 512, KD = 4096, ND = 4096, NS = 512;
constexpr int BM = 128, BN = 64, BK = 64, NST = 6;
constexpr int XB = BM * BK * 2, WB = BK * BN * 2, STB = XB + WB;
constexpr int NT = KD / BK;
constexpr int NWG = (MS / BM) * (ND / BN);
constexpr int WG_PER_DST = NWG / NR;
constexpr unsigned long long DEADLINE = 50000000ull;

struct Args {
    const unsigned short* x; const unsigned short* w;
    unsigned short* out[NR]; unsigned* ws[NR];
    int rank;
};

__device__ __forceinline__ unsigned cvt_pk(float lo, float hi) {
    unsigned r; asm volatile("v_cvt_pk_bf16_f32 %0, %1, %2" : "=v"(r) : "v"(lo), "v"(hi)); return r; }

__global__ __launch_bounds__(256, 1) void gemm_a2a(Args a) {
    __shared__ __attribute__((aligned(16))) unsigned char smem[NST * STB];
    const int tid = threadIdx.x, lane = tid & 63, wid = __builtin_amdgcn_readfirstlane(tid >> 6);
    const int bid = blockIdx.x;
    const int dst = bid & 7, sub = bid >> 3;
    const int tm = sub & 3, tn = sub >> 2;
    const int m0 = tm * BM, n0 = dst * NS + tn * BN;
    unsigned* myws = a.ws[a.rank];
    unsigned ticket = 0;
    if (tid == 0) ticket = __hip_atomic_fetch_add(myws, 1u, __ATOMIC_RELAXED, __HIP_MEMORY_SCOPE_AGENT);

    const int r8 = lane >> 3, c8 = lane & 7;
    unsigned xv[4], wv[2];
#pragma unroll
    for (int i = 0; i < 4; ++i) { const int m = (wid * 4 + i) * 8 + r8; xv[i] = (unsigned)((m0 + m) * KD * 2 + ((c8 ^ ((m >> 1) & 7)) << 4)) - i * 1024; }
#pragma unroll
    for (int i = 0; i < 2; ++i) { const int k = (wid * 2 + i) * 8 + r8; const int f = ((k >> 1) & 1) | (((k >> 3) & 1) << 1);
        wv[i] = (unsigned)(k * ND * 2 + n0 * 2 + ((c8 ^ (f << 1)) << 4)) - i * 1024; }
    u32x4 xrs, wrs;
    { const unsigned long long xa = (unsigned long long)a.x, wa = (unsigned long long)a.w;
      xrs.x = (unsigned)xa; xrs.y = (unsigned)(xa >> 32); xrs.z = MS * KD * 2; xrs.w = 0x00020000;
      wrs.x = (unsigned)wa; wrs.y = (unsigned)(wa >> 32); wrs.z = (unsigned)KD * ND * 2; wrs.w = 0x00020000; }
    const unsigned lds0 = (unsigned)(size_t)(LAS unsigned char*)smem;
    auto stage = [&](int t, int slot) {
        const unsigned mx = lds0 + slot * STB + wid * 4096, mw = lds0 + slot * STB + XB + wid * 2048;
        const unsigned sx = t * BK * 2, sw = (unsigned)t * BK * ND * 2;
        asm volatile("s_mov_b32 m0, %0\n\ts_nop 0\n\t"
                     "buffer_load_dwordx4 %1, %5, %6 offen lds\n\t"
                     "buffer_load_dwordx4 %2, %5, %6 offen offset:1024 lds\n\t"
                     "buffer_load_dwordx4 %3, %5, %6 offen offset:2048 lds\n\t"
                     "buffer_load_dwordx4 %4, %5, %6 offen offset:3072 lds"
                     :: "s"(mx), "v"(xv[0]), "v"(xv[1]), "v"(xv[2]), "v"(xv[3]), "s"(xrs), "s"(sx) : "memory");
        asm volatile("s_mov_b32 m0, %0\n\ts_nop 0\n\t"
                     "buffer_load_dwordx4 %1, %3, %4 offen lds\n\t"
                     "buffer_load_dwordx4 %2, %3, %4 offen offset:1024 lds"
                     :: "s"(mw), "v"(wv[0]), "v"(wv[1]), "s"(wrs), "s"(sw) : "memory");
    };
    const int wm = wid >> 1, wn = wid & 1, fr = lane & 15, g = lane >> 4, q = (lane & 15) >> 2, p = lane & 3;
    int xoff[2], woff[2];
#pragma unroll
    for (int ks = 0; ks < 2; ++ks) { const int m = wm * 64 + fr; xoff[ks] = m * 128 + (((4 * ks + g) ^ ((m >> 1) & 7)) << 4); }
#pragma unroll
    for (int ni = 0; ni < 2; ++ni) { const int k = 8 * g + q; const int f = ((k >> 1) & 1) | (((k >> 3) & 1) << 1);
        woff[ni] = XB + k * 128 + (((wn * 2 + ni) ^ f) << 5) + 8 * p; }
    struct Fr { bf16x8 x[4][2]; bf16x8 w[2][2]; };
    auto rd = [&](Fr& F, int slot) {
        const unsigned char* sb = smem + slot * STB;
#pragma unroll
        for (int ks = 0; ks < 2; ++ks) {
#pragma unroll
            for (int ni = 0; ni < 2; ++ni) {
                bf16x4 lo = __builtin_amdgcn_ds_read_tr16_b64_v4i16((LAS bf16x4*)(sb + woff[ni] + ks * 32 * 128));
                bf16x4 hi = __builtin_amdgcn_ds_read_tr16_b64_v4i16((LAS bf16x4*)(sb + woff[ni] + ks * 32 * 128 + 4 * 128));
                F.w[ni][ks] = __builtin_shufflevector(lo, hi, 0, 1, 2, 3, 4, 5, 6, 7);
            }
#pragma unroll
            for (int mi = 0; mi < 4; ++mi) F.x[mi][ks] = *(const bf16x8*)(sb + xoff[ks] + mi * 16 * 128);
        }
    };
    f32x4 acc[4][2] = {};
    auto mma = [&](const Fr& F) {
#pragma unroll
        for (int ks = 0; ks < 2; ++ks)
#pragma unroll
            for (int mi = 0; mi < 4; ++mi)
#pragma unroll
                for (int ni = 0; ni < 2; ++ni)
                    acc[mi][ni] = __builtin_amdgcn_mfma_f32_16x16x32_bf16(F.w[ni][ks], F.x[mi][ks], acc[mi][ni], 0, 0, 0);
    };
#define WAITV(n) asm volatile("s_waitcnt vmcnt(" #n ")" ::: "memory")
#define WAITL0 __builtin_amdgcn_s_waitcnt(0xC07F)
#define BAR do { asm volatile("" ::: "memory"); __builtin_amdgcn_s_barrier(); asm volatile("" ::: "memory"); } while (0)
    Fr F0, F1;
#pragma unroll
    for (int s = 0; s < NST; ++s) stage(s, s);
    WAITV(30); BAR;
    rd(F0, 0);
    int slot = 0;
    auto step = [&](int i, Fr& Fc, Fr& Fn) {
        WAITV(24); WAITL0; BAR;
        int nslot = slot + 1; if (nslot == NST) nslot = 0;
        int tn_ = i + NST; if (tn_ > NT - 1) tn_ = NT - 1;
        stage(tn_, slot);
        rd(Fn, nslot);
        __builtin_amdgcn_s_setprio(1); mma(Fc); __builtin_amdgcn_s_setprio(0);
        slot = nslot;
    };
    for (int i = 0; i < NT - 2; i += 2) { step(i, F0, F1); step(i + 1, F1, F0); }
    step(NT - 2, F0, F1);
    WAITL0; mma(F1);
    WAITV(0); BAR;
    constexpr int ES = 144;
#pragma unroll
    for (int mi = 0; mi < 4; ++mi)
#pragma unroll
        for (int ni = 0; ni < 2; ++ni) {
            u32x2 v; v.x = cvt_pk(acc[mi][ni][0], acc[mi][ni][1]); v.y = cvt_pk(acc[mi][ni][2], acc[mi][ni][3]);
            *(u32x2*)(smem + (wm * 64 + mi * 16 + fr) * ES + (wn * 32 + ni * 16 + 4 * g) * 2) = v;
        }
    WAITL0; BAR;
    unsigned short* obase = a.out[dst] + (size_t)(a.rank * MS + m0) * NS + tn * BN;
#pragma unroll
    for (int i = 0; i < 4; ++i) {
        const int row = i * 32 + (tid >> 3), ch = tid & 7;
        const u32x4 v = *(const u32x4*)(smem + row * ES + ch * 16);
        *(u32x4*)(obase + (size_t)row * NS + ch * 8) = v;
    }
    WAITV(0);
    __syncthreads();
    if (tid == 0) {
        const unsigned call = ticket / NWG + 1;
        if (dst != a.rank) {
            __builtin_amdgcn_fence(__ATOMIC_RELEASE, "");
            asm volatile("s_waitcnt vmcnt(0)" ::: "memory");
            __hip_atomic_fetch_add(a.ws[dst] + 64 + 32 * a.rank, 1u, __ATOMIC_RELAXED, __HIP_MEMORY_SCOPE_SYSTEM);
        } else if (sub == 0) {
            const unsigned long long t0 = __builtin_amdgcn_s_memrealtime();
            bool bad = false;
            for (int s = 0; s < NR; ++s) {
                if (s == a.rank) continue;
                while (__hip_atomic_load(myws + 64 + 32 * s, __ATOMIC_RELAXED, __HIP_MEMORY_SCOPE_SYSTEM) < call * WG_PER_DST) {
                    if (__builtin_amdgcn_s_memrealtime() - t0 > DEADLINE) { bad = true; break; }
                    __builtin_amdgcn_s_sleep(2);
                }
                if (bad) break;
            }
            if (bad) { __hip_atomic_store(myws + 32, call, __ATOMIC_RELAXED, __HIP_MEMORY_SCOPE_AGENT); obase[0] = 0x7fc0; }
        }
    }
}
}

extern "C" void kernel_launch(int rank, int nranks, ncclComm_t comm, hipStream_t stream,
                              void* const* d_in, const int* in_sizes, int n_in,
                              void* d_out, int out_size, const int* in_dtypes, int out_dtype,
                              void* const* d_out_all, void* const* d_ws_all, size_t ws_bytes) {
    if (nranks != NR || (unsigned)rank >= (unsigned)NR) return;
    Args a;
    a.x = (const unsigned short*)d_in[0]; a.w = (const unsigned short*)d_in[1]; a.rank = rank;
    for (int p = 0; p < NR; ++p) { a.out[p] = (unsigned short*)d_out_all[p]; a.ws[p] = (unsigned*)d_ws_all[p]; }
    gemm_a2a<<<NWG, 256, 0, stream>>>(a);
    HIP_CHECK(hipGetLastError());
}
```

```cpp
#include <hip/hip_runtime.h>
#include <rccl/rccl.h>
#include <cstdio>
#include <cstdint>
#include <type_traits>

#define LSLEEP 2
#define BAL 0
#define LPRIO 0
#define RPRIO 0
#define CPRIO 0
#define RDELAY 24
#define NTR -1
#define NTF 5
#define NTAUX 2
#define FRQ 6
#define STAMPS 0
#define ABL 0
#define CT(tj, d) ((unsigned)((tj) + (d)) & 7u)

namespace {
typedef short bf16x8 __attribute__((ext_vector_type(8)));
typedef short bf16x4 __attribute__((ext_vector_type(4)));
typedef float f32x4 __attribute__((ext_vector_type(4)));
typedef unsigned u32x4 __attribute__((ext_vector_type(4)));
typedef unsigned u32x2 __attribute__((ext_vector_type(2)));
#define LAS __attribute__((address_space(3)))

constexpr int NR = 8, MS = 512, KD = 4096, ND = 4096, NS = 512;
constexpr int NWG = 256;
constexpr unsigned long long DEADLINE = 50000000ull;
constexpr unsigned EX_BYTES = 32u << 20;
constexpr unsigned RF_OFF = EX_BYTES + 65536;
constexpr unsigned CNT_OFF = EX_BYTES + 131072;
constexpr unsigned TMO_OFF = EX_BYTES + 196608;
constexpr unsigned DBG_OFF = EX_BYTES + 262144;
constexpr unsigned WS_NEED = EX_BYTES + (1u << 20);
constexpr int OUT_RS = 144, OUT_B = 16 * OUT_RS;
constexpr int L_W = 0, L_X = 65536;
constexpr int L_OUT = 131072, L_FLAG = L_OUT + 8 * OUT_B;
constexpr int L_FULL = L_FLAG, L_FREE = L_FLAG + 256, L_ORDY = L_FLAG + 5 * 256, L_END = L_FLAG + 7 * 256;
static_assert(L_X + 4 * 5 * 4096 <= L_FLAG && L_END <= 163840, "lds map");

struct Args {
    const unsigned short* x; const unsigned short* w;
    unsigned char* out[NR]; unsigned char* ws[NR];
    int rank;
};

template <int I, int N, class F> __device__ __forceinline__ void sfor(F&& f) { if constexpr (I < N) { f(std::integral_constant<int, I>{}); sfor<I + 1, N>(f); } }
template <int N> __device__ __forceinline__ void waitv() { asm volatile("s_waitcnt vmcnt(%0)" :: "n"(N) : "memory"); }
template <int N> __device__ __forceinline__ void waitl() { asm volatile("s_waitcnt lgkmcnt(%0)" :: "n"(N) : "memory"); }
__device__ __forceinline__ unsigned cvt_pk(float lo, float hi) { unsigned r; asm volatile("v_cvt_pk_bf16_f32 %0, %1, %2" : "=v"(r) : "v"(lo), "v"(hi)); return r; }
__device__ __forceinline__ unsigned tag_in(unsigned v, unsigned bit, unsigned parm) { unsigned r; asm volatile("v_bfi_b32 %0, %1, %2, %3" : "=v"(r) : "s"(bit), "v"(parm), "v"(v)); return r; }
__device__ __forceinline__ u32x4 mk_rsrc(const void* p, unsigned bytes) { const unsigned long long a = (unsigned long long)p; u32x4 r; r.x = (unsigned)a; r.y = (unsigned)(a >> 32); r.z = bytes; r.w = 0x00020000; return r; }
__device__ __forceinline__ unsigned lds_rd(unsigned addr) { unsigned v; asm volatile("ds_read_b32 %0, %1\n\ts_waitcnt lgkmcnt(0)" : "=v"(v) : "v"(addr) : "memory"); return v; }
__device__ __forceinline__ void lds_wr(unsigned addr, unsigned v) { asm volatile("ds_write_b32 %0, %1" :: "v"(addr), "v"(v) : "memory"); }
__device__ __forceinline__ unsigned rfl(unsigned v) { return __builtin_amdgcn_readfirstlane(v); }
template <int SL> __device__ __forceinline__ void lds_wait(unsigned addr, unsigned need) { while (__any(lds_rd(addr) < need)) { if (SL) __builtin_amdgcn_s_sleep(SL); } }
__device__ __forceinline__ int dest_of(int r, int tj, int rank) { if (BAL) return (rank + 1 + tj + r) & 7; int t = tj + r; t = t >= 7 ? t - 7 : t; t = t >= 7 ? t - 7 : t; return r == 7 ? rank : (rank + 1 + t) & 7; }
__device__ __forceinline__ unsigned slot_off(int o, int k, unsigned par) { return (unsigned)(par ? k * 8 + o : o * 8 + k) * 2048u; }
__device__ __forceinline__ unsigned slot_bit(int o, int k, unsigned par) { return (par ? k < o : o < k) ? 1u : 0x10000u; }

#define STAMP(i) do {} while (0)
#define STAMPC(i) do {} while (0)
#define STAMP_OUT() do {} while (0)

__global__ __launch_bounds__(512, 2) void gemm_a2a(Args a) {
    __shared__ __attribute__((aligned(16))) unsigned char smem[L_END];
    const int tid = threadIdx.x, lane = tid & 63, wid = __builtin_amdgcn_readfirstlane(tid >> 6);
    const int bid = blockIdx.x, kq = bid & 7, tile = bid >> 3, tm = tile & 3, tj = tile >> 2;
    const int rot = tj + 2 * tm;
    const unsigned lds0 = (unsigned)(size_t)(LAS unsigned char*)smem;
    unsigned char* const myws = a.ws[a.rank];
    const int r8 = lane >> 3, c8 = lane & 7;
    unsigned stampv = 0; (void)stampv;

    if (wid < 4) {
        STAMP(0);
        const u32x4 wsr = mk_rsrc(myws, WS_NEED);
        unsigned callv;
        asm volatile("buffer_load_dword %0, off, %1, %2 sc1" : "=v"(callv) : "s"(wsr), "s"(CNT_OFF + bid * 64) : "memory");
        const u32x4 xr = mk_rsrc(a.x, MS * KD * 2);
        unsigned xv[4];
#pragma unroll
        for (int j = 0; j < 4; ++j) { const int m = j * 8 + r8; xv[j] = (unsigned)((tm * 128 + wid * 32 + m) * KD * 2 + ((c8 ^ ((m >> 1) & 7)) << 4)) - j * 1024; }
        const unsigned xl = lds0 + L_X + wid * 20480;
        auto xload = [&](int i) {
            asm volatile("s_mov_b32 m0, %0\n\ts_nop 0\n\t"
                         "buffer_load_dwordx4 %1, %5, %6 offen lds\n\t"
                         "buffer_load_dwordx4 %2, %5, %6 offen offset:1024 lds\n\t"
                         "buffer_load_dwordx4 %3, %5, %6 offen offset:2048 lds\n\t"
                         "buffer_load_dwordx4 %4, %5, %6 offen offset:3072 lds"
                         :: "s"(xl + (i % 5) * 4096), "v"(xv[0]), "v"(xv[1]), "v"(xv[2]), "v"(xv[3]), "s"(xr), "s"((unsigned)(kq * 1024 + ((i + rot) & 7) * 128)) : "memory");
        };
        xload(0); xload(1);
        asm volatile("s_barrier" ::: "memory");
        xload(2); xload(3); xload(4);
        const int fr = lane & 15, g = lane >> 4, q4 = (lane & 15) >> 2, p4 = lane & 3;
        bf16x8 xf[16][2];
        unsigned wa[4], xa[2];
#pragma unroll
        for (int ni = 0; ni < 4; ++ni) { const int k = 8 * g + q4; const int f = ((k >> 1) & 1) | (((k >> 3) & 1) << 1); wa[ni] = lds0 + L_W + k * 128 + ((ni ^ f) << 5) + 8 * p4; }
#pragma unroll
        for (int ks = 0; ks < 2; ++ks) xa[ks] = xl + fr * 128 + (((4 * ks + g) ^ ((fr >> 1) & 7)) << 4);
        unsigned fulla = lds0 + L_FULL, freea = lds0 + L_FREE + wid * 256 + lane * 4, exv = lane * 16;
        f32x4 acc[2][4];
#pragma unroll
        for (int ni = 0; ni < 4; ++ni) { asm volatile("" : "=v"(acc[0][ni]), "=v"(acc[1][ni]), "+v"(wa[ni])); }
        asm volatile("" : "+v"(fulla), "+v"(freea), "+v"(exv));
        bf16x4 wf[8][2];
        auto rdw = [&](int qq, int h) __attribute__((always_inline)) {
            const int q = qq & 63;
            asm volatile("ds_read_b64_tr_b16 %0, %1 offset:%2" : "=v"(wf[q & 7][h]) : "v"(wa[q & 3]), "n"((q >> 3) * 8192 + (((q >> 2) & 1) * 32 + h * 4) * 128));
        };
        unsigned exo[2], tb[2], parm;
#pragma unroll
        for (int i = 0; i < 8; ++i) {
            if (i <= 3) waitv<16>(); else if (i == 4) waitv<12>(); else if (i == 5) waitv<8>(); else if (i == 6) waitv<4>(); else waitv<0>();
#pragma unroll
            for (int ks = 0; ks < 2; ++ks)
#pragma unroll
                for (int mi = 0; mi < 2; ++mi)
                    asm volatile("ds_read_b128 %0, %1 offset:%2" : "=v"(xf[2 * i + ks][mi]) : "v"(xa[ks]), "n"((i % 5) * 4096 + mi * 2048));
            if (i < 3) { waitl<0>(); xload(i + 5); }
            if (i == 3) {
                asm volatile("" : "+v"(callv));
                const unsigned par = (rfl(callv) + 1) & 1;
                parm = par * 0x10001u; asm volatile("" : "+v"(parm));
#pragma unroll
                for (int mi = 0; mi < 2; ++mi) { exo[mi] = rfl(ABL == 1 ? WS_NEED : slot_off(2 * wid + mi, kq, par)); tb[mi] = rfl(slot_bit(2 * wid + mi, kq, par)); asm volatile("" : "+s"(exo[mi]), "+s"(tb[mi])); }
            }
        }
        STAMP(1);
        waitl<0>();
        unsigned freev = 1;
        lds_wr(freea, freev);
        lds_wait<0>(fulla, 1);
        __builtin_amdgcn_s_setprio(CPRIO);
#pragma unroll
        for (int q = 0; q < 7; ++q) { rdw(q, 0); rdw(q, 1); }
        STAMP(2); STAMPC(40);
        u32x4 pk[2];
        unsigned fv = 0; unsigned long long fok = 0;
        unsigned long long seg[9]; (void)seg;
#pragma unroll 1
        for (int r = 0; r < 8; ++r) {
            const unsigned flip = (r & 1) ? 0xffff0000u : 0x10000u;
            const unsigned exs = r == 0 ? WS_NEED : (unsigned)((r - 1) * 32 + tile) * 131072u;
            sfor<0, 64>([&](auto Q) __attribute__((always_inline)) {
                constexpr int q = Q, ni = q & 3, s = q >> 2;
                if (!(q & 1)) waitl<10>();
                if ((q & 31) == 24)
                    asm volatile("v_cmp_gt_u32 %0, %1, %2" : "=s"(fok) : "s"(2 * r + (q >> 5) + 2), "v"(fv));
                if ((q & 31) == 25) {
                    asm volatile("" : "+s"(fok));
                    if (__builtin_expect(fok != 0, 0)) {
                        lds_wait<0>(fulla, 2 * r + (q >> 5) + 2);
                    }
                }
                if (q >= 57 && q < 61) { wa[(q + 7) & 3] += flip; asm volatile("" : "+v"(wa[(q + 7) & 3])); }
                const bf16x8 wq = __builtin_shufflevector(wf[q & 7][0], wf[q & 7][1], 0, 1, 2, 3, 4, 5, 6, 7);
#pragma unroll
                for (int mi = 0; mi < 2; ++mi) {
                    if constexpr (q < 4) {
                        pk[mi][2 * (q & 1)] = cvt_pk(acc[mi][ni][0], acc[mi][ni][1]); pk[mi][2 * (q & 1) + 1] = cvt_pk(acc[mi][ni][2], acc[mi][ni][3]);
                        asm volatile("v_mfma_f32_16x16x32_bf16 %0, %1, %2, 0" : "+v"(acc[mi][ni]) : "v"(wq), "v"(xf[s][mi]));
                        if (!(q & 1)) pk[mi][0] = tag_in(pk[mi][0], tb[mi], parm);
                        else asm volatile("buffer_store_dwordx4 %0, %1, %2, %3 offen offset:%4 sc1\n\ts_nop 0" :: "v"(pk[mi]), "v"(exv), "s"(wsr), "s"(exs + exo[mi]), "n"((q >> 1) * 1024) : "memory");
                    } else
                        asm volatile("v_mfma_f32_16x16x32_bf16 %0, %1, %2, %0" : "+v"(acc[mi][ni]) : "v"(wq), "v"(xf[s][mi]));
                    rdw(q + 7, mi);
                }
                if (q == FRQ) lds_wr(freea, freev);
                if ((q & 31) == 16) asm volatile("ds_read_b32 %0, %1" : "=v"(fv) : "v"(fulla) : "memory");
                if (q == 30) { freev += 1; asm volatile("" : "+v"(freev)); lds_wr(freea, freev); }
                if (q == 40) { freev += 1; asm volatile("" : "+v"(freev)); }
            });
        }
        asm volatile("s_waitcnt lgkmcnt(0)\n\ts_nop 7\n\ts_nop 7" ::: "memory");
        const unsigned exs = (unsigned)(7 * 32 + tile) * 131072u;
#pragma unroll
        for (int mi = 0; mi < 2; ++mi)
#pragma unroll
            for (int h = 0; h < 2; ++h) {
                u32x4 v; v[0] = tag_in(cvt_pk(acc[mi][2 * h][0], acc[mi][2 * h][1]), tb[mi], parm); v[1] = cvt_pk(acc[mi][2 * h][2], acc[mi][2 * h][3]);
                v[2] = cvt_pk(acc[mi][2 * h + 1][0], acc[mi][2 * h + 1][1]); v[3] = cvt_pk(acc[mi][2 * h + 1][2], acc[mi][2 * h + 1][3]);
                asm volatile("buffer_store_dwordx4 %0, %1, %2, %3 offen offset:%4 sc1\n\ts_nop 1" :: "v"(v), "v"(exv), "s"(wsr), "s"(exs + exo[mi]), "n"(h * 1024) : "memory");
            }
        lds_wr(freea, freev);
        STAMP(15); STAMPC(48);
        waitl<0>();
        STAMP_OUT();
        return;
    }
    if (wid == 4) {
        const u32x4 wr = mk_rsrc(a.w, (unsigned)KD * ND * 2);
        unsigned wv[8];
#pragma unroll
        for (int j = 0; j < 8; ++j) { const int f = ((r8 >> 1) & 1) | ((j & 1) << 1); wv[j] = (unsigned)((kq * 512 + j * 8 + r8) * ND * 2 + ((c8 ^ (f << 1)) << 4)) - (j & 3) * 1024; }
        auto issue = [&](int i, unsigned co, unsigned lo) {
            const unsigned so = (unsigned)((i + rot) & 7) * (64u * ND * 2) + co, m = lds0 + L_W + i * 8192 + lo;
            asm volatile("s_mov_b32 m0, %0\n\ts_nop 0\n\t"
                         "buffer_load_dwordx4 %2, %10, %11 offen lds\n\t"
                         "buffer_load_dwordx4 %3, %10, %11 offen offset:1024 lds\n\t"
                         "buffer_load_dwordx4 %4, %10, %11 offen offset:2048 lds\n\t"
                         "buffer_load_dwordx4 %5, %10, %11 offen offset:3072 lds\n\t"
                         "s_mov_b32 m0, %1\n\ts_nop 0\n\t"
                         "buffer_load_dwordx4 %6, %10, %11 offen lds\n\t"
                         "buffer_load_dwordx4 %7, %10, %11 offen offset:1024 lds\n\t"
                         "buffer_load_dwordx4 %8, %10, %11 offen offset:2048 lds\n\t"
                         "buffer_load_dwordx4 %9, %10, %11 offen offset:3072 lds"
                         :: "s"(m), "s"(m + 4096), "v"(wv[0]), "v"(wv[1]), "v"(wv[2]), "v"(wv[3]), "v"(wv[4]), "v"(wv[5]), "v"(wv[6]), "v"(wv[7]), "s"(wr), "s"(so) : "memory");
        };
        __builtin_amdgcn_s_setprio(LPRIO);
        const unsigned fulla = lds0 + L_FULL + lane * 4, freea = lds0 + L_FREE + (lane & 3) * 256;
#pragma unroll 1
        for (int gi = 0; gi < 16; ++gi) {
            const int r = gi >> 1, d = dest_of(r, tj, a.rank); const unsigned co = (unsigned)d * (NS * 2) + CT(tj, d) * 128;
            if (gi >= 2) lds_wait<LSLEEP>(freea, gi < 4 ? 1u : (unsigned)gi - 2);
            const int i = (gi & 1) * 4;
            issue(i, co, (gi & 2) * 32768);
            if (gi == 0) asm volatile("s_barrier" ::: "memory");
            issue(i + 1, co, (gi & 2) * 32768);
            waitv<16>(); lds_wr(fulla, (unsigned)gi);
            issue(i + 2, co, (gi & 2) * 32768); issue(i + 3, co, (gi & 2) * 32768);
            if (gi & 1) STAMP(r);
        }
        waitv<0>(); lds_wr(fulla, 17u);
        STAMP(8);
        waitl<0>();
        STAMP_OUT();
        return;
    }
    if (wid == 7) {
        for (int i = lane; i < (L_END - L_FLAG) / 4; i += 64) lds_wr(lds0 + L_FLAG + i * 4, 0u);
        waitl<0>();
    }
    asm volatile("s_barrier" ::: "memory");
    const unsigned call = rfl(__hip_atomic_load((unsigned*)(myws + CNT_OFF + bid * 64), __ATOMIC_RELAXED, __HIP_MEMORY_SCOPE_AGENT)) + 1;
    const unsigned long long t0 = __builtin_amdgcn_s_memrealtime();
    unsigned* const tmo = (unsigned*)(myws + TMO_OFF);
    __builtin_amdgcn_s_setprio(RPRIO);
    if (wid >= 6) {
        const __amdgpu_buffer_rsrc_t rs = __builtin_amdgcn_make_buffer_rsrc(myws, 0, ABL == 1 ? 0 : EX_BYTES, 0x00020000);
        const int fr = lane & 15, g = lane >> 4;
        const unsigned par = call & 1, parm = par * 0x10001u;
#pragma unroll 1
        for (int r = wid - 6; r < 8; r += 2) {
            lds_wait<1>(lds0 + L_FREE + (lane & 3) * 256, 2u * r + 3);
            STAMP(r);
            if (RDELAY) __builtin_amdgcn_s_sleep(RDELAY);
            const unsigned eo = (unsigned)(r * 32 + tile) * 131072u;
            u32x4 v[8][2];
            bool bad = false; unsigned pend = 0xff;
            for (unsigned n = 1;; ++n) {
#pragma unroll
                for (int s = 0; s < 8; ++s)
                    if (pend & (1u << s)) {
#pragma unroll
                        for (int h = 0; h < 2; ++h) v[s][h] = __builtin_amdgcn_raw_buffer_load_b128(rs, lane * 16 + h * 1024, eo + slot_off(kq, s, par), 16);
                    }
#pragma unroll
                for (int s = 0; s < 8; ++s) if (ABL == 1 || __all((((v[s][0][0] ^ parm) | (v[s][1][0] ^ parm)) & slot_bit(kq, s, par)) == 0)) pend &= ~(1u << s);
                if (pend == 0) break;
                if ((n & 63) == 0 && (rfl(__hip_atomic_load(tmo, __ATOMIC_RELAXED, __HIP_MEMORY_SCOPE_AGENT)) == call || __builtin_amdgcn_s_memrealtime() - t0 > DEADLINE)) { bad = true; break; }
            }
            if (bad) __hip_atomic_store(tmo, call, __ATOMIC_RELAXED, __HIP_MEMORY_SCOPE_AGENT);
            STAMP(8 + r);
            unsigned char* const ob = smem + L_OUT + r * OUT_B + fr * OUT_RS + g * 8;
#pragma unroll
            for (int h = 0; h < 2; ++h) {
                float lo[4] = {0.f, 0.f, 0.f, 0.f}, hi[4] = {0.f, 0.f, 0.f, 0.f};
#pragma unroll
                for (int s = 0; s < 8; ++s) {
                    const bool tl = slot_bit(kq, s, par) == 1u;
                    lo[0] += __uint_as_float(((v[s][h][0] << 16) & (tl ? 0xfffe0000u : 0xffff0000u)) | (tl ? 0x8000u : 0u));
                    hi[0] += __uint_as_float((v[s][h][0] & (tl ? 0xffff0000u : 0xfffe0000u)) | (tl ? 0u : 0x8000u));
#pragma unroll
                    for (int e = 1; e < 4; ++e) { lo[e] += __uint_as_float(v[s][h][e] << 16); hi[e] += __uint_as_float(v[s][h][e] & 0xffff0000u); }
                }
                u32x2 o0, o1; o0.x = cvt_pk(lo[0], hi[0]); o0.y = cvt_pk(lo[1], hi[1]); o1.x = cvt_pk(lo[2], hi[2]); o1.y = cvt_pk(lo[3], hi[3]);
                if (bad) { o0.x = o0.y = o1.x = o1.y = 0x7fc07fc0u; }
                *(u32x2*)(ob + h * 64) = o0; *(u32x2*)(ob + h * 64 + 32) = o1;
            }
            asm volatile("" ::: "memory");
            lds_wr(lds0 + L_ORDY + (r & 1) * 256 + lane * 4, (unsigned)r + 1);
            STAMP(16 + r);
        }
        waitl<0>();
        STAMP_OUT();
        return;
    }
    const unsigned ov = (unsigned)((a.rank * MS + tm * 128 + kq * 16 + r8) * NS * 2 + c8 * 16);
#pragma unroll 1
    for (int r = 0; r < 8; ++r) {
        const int d = dest_of(r, tj, a.rank);
        const __amdgpu_buffer_rsrc_t ro = __builtin_amdgcn_make_buffer_rsrc(a.out[d & 7], 0, (ABL == 2 || ABL == 1) && d != a.rank ? 0 : NR * MS * NS * 2, 0x00020000);
        lds_wait<1>(lds0 + L_ORDY + (r & 1) * 256, (unsigned)r + 1);
        const unsigned char* ob = smem + L_OUT + r * OUT_B + r8 * OUT_RS + c8 * 16;
        const u32x4 v0 = *(const u32x4*)ob, v1 = *(const u32x4*)(ob + 8 * OUT_RS);
        if (r <= NTR) {
            __builtin_amdgcn_raw_buffer_store_b128(v0, ro, ov, CT(tj, d) * 128u, NTAUX);
            __builtin_amdgcn_raw_buffer_store_b128(v1, ro, ov, CT(tj, d) * 128u + 8u * NS * 2, NTAUX);
        } else {
            __builtin_amdgcn_raw_buffer_store_b128(v0, ro, ov, CT(tj, d) * 128u, 17);
            __builtin_amdgcn_raw_buffer_store_b128(v1, ro, ov, CT(tj, d) * 128u + 8u * NS * 2, 17);
        }
        STAMP(r);
        if (NTR >= 0 && r == NTF) { waitv<0>(); asm volatile("buffer_wbl2 sc0 sc1" ::: "memory"); waitv<0>(); STAMP(10); }
        if (r == (BAL ? 7 : 6)) {
            waitv<0>();
            STAMP(8);
#pragma unroll 1
            for (int p = 0; p < NR; ++p)
                if (p != a.rank && lane == 0) __hip_atomic_store((unsigned*)(a.ws[p] + RF_OFF + bid * 128) + a.rank, call, __ATOMIC_RELAXED, __HIP_MEMORY_SCOPE_SYSTEM);
        }
    }
    {
        unsigned* const rf = (unsigned*)(myws + RF_OFF + bid * 128) + (lane & 7);
        bool bad = false;
        for (unsigned n = 1;; ++n) {
            const unsigned f = __hip_atomic_load(rf, __ATOMIC_RELAXED, __HIP_MEMORY_SCOPE_SYSTEM);
            if (__all((lane & 7) == a.rank || f >= call)) break;
            if ((n & 63) == 0 && (__hip_atomic_load(tmo, __ATOMIC_RELAXED, __HIP_MEMORY_SCOPE_AGENT) == call || __builtin_amdgcn_s_memrealtime() - t0 > DEADLINE)) { bad = true; break; }
            __builtin_amdgcn_s_sleep(1);
        }
        if (lane == 0) {
            if (bad) { __hip_atomic_store(tmo, call, __ATOMIC_RELAXED, __HIP_MEMORY_SCOPE_AGENT); *(unsigned short*)(a.out[a.rank] + ov + CT(tj, a.rank) * 128) = 0x7fc0; }
            __hip_atomic_store((unsigned*)(myws + CNT_OFF + bid * 64), call, __ATOMIC_RELAXED, __HIP_MEMORY_SCOPE_AGENT);
        }
        STAMP(9);
        STAMP_OUT();
    }
}
}

extern "C" void kernel_launch(int rank, int nranks, ncclComm_t comm, hipStream_t stream,
                              void* const* d_in, const int* in_sizes, int n_in,
                              void* d_out, int out_size, const int* in_dtypes, int out_dtype,
                              void* const* d_out_all, void* const* d_ws_all, size_t ws_bytes) {
    if (nranks != NR || (unsigned)rank >= (unsigned)NR || ws_bytes < WS_NEED || n_in < 2 || in_sizes[0] != MS * KD || in_sizes[1] != KD * ND || out_size != NR * MS * NS) {
        fprintf(stderr, "kernel_launch: unexpected problem shape\n"); return; }
    Args a;
    a.x = (const unsigned short*)d_in[0]; a.w = (const unsigned short*)d_in[1]; a.rank = rank;
    for (int p = 0; p < NR; ++p) { a.out[p] = (unsigned char*)d_out_all[p]; a.ws[p] = (unsigned char*)d_ws_all[p]; }
    gemm_a2a<<<NWG, 512, 0, stream>>>(a);
    hipError_t e = hipGetLastError();
    if (e != hipSuccess) fprintf(stderr, "HIP error %s\n", hipGetErrorString(e));
}
```

```cpp
#include <hip/hip_runtime.h>
#include <rccl/rccl.h>
#include <cstdio>
#include <cstdint>
#include <type_traits>

#define PRIO2 1
#define SPRIO 3
#define TOUCH 0
#define EARLY 0
#define PACE 2
#define IMPQ 54
#define SDELAY 8
#define TDELAY 20
#define APRIO0 1
#define W2DELAY 8
#define RETRY_LIMIT 600000u
#define MIDA 38
#define MIDB 40
#define C24I 0
#define RDELAY 10
#define STAMPS 0
#define ABL 0

namespace {
typedef short bf16x8 __attribute__((ext_vector_type(8)));
typedef short bf16x4 __attribute__((ext_vector_type(4)));
typedef float f32x4 __attribute__((ext_vector_type(4)));
typedef unsigned u32x4 __attribute__((ext_vector_type(4)));
typedef unsigned u32x2 __attribute__((ext_vector_type(2)));
#define LAS __attribute__((address_space(3)))

constexpr int NR = 8, MS = 512, KD = 4096, ND = 4096, NS = 512;
constexpr int NWG = 256;
constexpr unsigned long long DEADLINE = 50000000ull;
constexpr unsigned EX_BYTES = 16u << 20;
constexpr unsigned RF_OFF = EX_BYTES;
constexpr unsigned CNT_OFF = EX_BYTES + 131072;
constexpr unsigned TMO_OFF = EX_BYTES + 196608;
constexpr unsigned DBG_OFF = EX_BYTES + 262144;
constexpr unsigned WS_NEED = EX_BYTES + (1u << 20);
constexpr int OUT_RS = 144, OUT_B = 16 * OUT_RS;
constexpr int L_W = 0;
constexpr int L_HO = 131072, L_OWN = L_HO + 16384, L_OUT = L_OWN + 4096, L_END = L_OUT + 2 * OUT_B;
static_assert(L_END <= 163840, "lds map");

struct Args {
    const unsigned short* x; const unsigned short* w;
    unsigned char* out[NR]; unsigned char* ws[NR];
    int rank;
};

template <int I, int N, class F> __device__ __forceinline__ void sfor(F&& f) { if constexpr (I < N) { f(std::integral_constant<int, I>{}); sfor<I + 1, N>(f); } }
template <int N> __device__ __forceinline__ void waitv() { asm volatile("s_waitcnt vmcnt(%0)" :: "n"(N) : "memory"); }
template <int N> __device__ __forceinline__ void waitl() { asm volatile("s_waitcnt lgkmcnt(%0)" :: "n"(N) : "memory"); }
__device__ __forceinline__ unsigned cvt_pk(float lo, float hi) { unsigned r; asm("v_cvt_pk_bf16_f32 %0, %1, %2" : "=v"(r) : "v"(lo), "v"(hi)); return r; }
__device__ __forceinline__ u32x4 mk_rsrc(const void* p, unsigned bytes) { const unsigned long long a = (unsigned long long)p; u32x4 r; r.x = (unsigned)a; r.y = (unsigned)(a >> 32); r.z = bytes; r.w = 0x00020000; return r; }
__device__ __forceinline__ unsigned lds_rd(unsigned addr) { unsigned v; asm volatile("ds_read_b32 %0, %1\n\ts_waitcnt lgkmcnt(0)" : "=v"(v) : "v"(addr) : "memory"); return v; }
__device__ __forceinline__ void lds_wr(unsigned addr, unsigned v) { asm volatile("ds_write_b32 %0, %1" :: "v"(addr), "v"(v) : "memory"); }
__device__ __forceinline__ unsigned rfl(unsigned v) { return __builtin_amdgcn_readfirstlane(v); }
template <int SL> __device__ __forceinline__ void lds_wait(unsigned addr, unsigned need) { while (__any(lds_rd(addr) < need)) { if (SL) __builtin_amdgcn_s_sleep(SL); } }
__device__ __forceinline__ float bf_lo(unsigned v) { return __uint_as_float(v << 16); }
__device__ __forceinline__ float bf_hi(unsigned v) { return __uint_as_float(v & 0xffff0000u); }
__device__ __forceinline__ int tile_of(int r, int cg, int rank) {
    const int n = r < 3 ? r * 16 + cg : 48 + (cg >> 1), ct = (n * 37) >> 8;
    return r == 3 && (cg & 1) ? rank * 8 + (cg >> 1) : ((rank + 1 + n - 7 * ct) & 7) * 8 + ct;
}
__device__ __forceinline__ unsigned slot_off(int o, int k, unsigned par) { return (unsigned)(par ? k * 4 + o : o * 4 + k) * 4096u; }
__device__ __forceinline__ unsigned slot_bit(int o, int k, unsigned par) { return (par ? k < o : o < k) ? 1u : 0x10000u; }

#define STAMP(i) do {} while (0)
#define STAMP_OUT() do {} while (0)

#define BAR asm volatile("s_barrier" ::: "memory")
#define PIN_S(x) asm volatile("" : "+s"(x))
__global__ __launch_bounds__(512, 2) void gemm_a2a(Args a) {
    __shared__ __attribute__((aligned(16))) unsigned char smem[L_END];
    const int tid = threadIdx.x, bid = blockIdx.x;
    int wid = __builtin_amdgcn_readfirstlane(tid >> 6);
    int lane = tid & 63;
    #define G (wid >> 2)
#define w4 (wid & 3)
#define mi_ (wid & 1)
    const int kq = bid & 3, tm = (bid >> 3) & 3, cg = ((bid >> 2) & 1) * 8 + (bid >> 5);
    const int rot = (bid >> 5) + 2 * tm;
    const unsigned lds0 = rfl((unsigned)(size_t)(LAS unsigned char*)smem);
    unsigned char* const myws = a.ws[a.rank];
#define r8 (lane >> 3)
#define c8 (lane & 7)
#define fr (lane & 15)
#define g4 (lane >> 4)
    unsigned stampv = 0; (void)stampv;
    STAMP(0);
    if (G == 1) __builtin_amdgcn_s_setprio(SPRIO);

    const u32x4 wr = mk_rsrc(a.w, (unsigned)KD * ND * 2);
    unsigned wv[4];
#pragma unroll
    for (int j = 0; j < 4; ++j) { const int f = ((r8 >> 1) & 1) | ((j & 1) << 1); wv[j] = (unsigned)((j * 8 + r8) * ND * 2 + ((c8 ^ (f << 1)) << 4)) - j * 1024; }
    auto krow = [&](int gg, int i) { return (unsigned)(kq * 1024 + gg * 512 + ((i + rot) & 7) * 64) * (ND * 2); };
    auto slot = [&](int gg, int i) { return lds0 + L_W + gg * 65536 + i * 8192; };
    auto dma = [&](unsigned so, unsigned m) __attribute__((always_inline)) {
        asm volatile("s_mov_b32 m0, %0\n\ts_nop 0\n\t"
                     "buffer_load_dwordx4 %2, %6, %7 offen lds\n\t"
                     "buffer_load_dwordx4 %3, %6, %7 offen offset:1024 lds\n\t"
                     "buffer_load_dwordx4 %4, %6, %7 offen offset:2048 lds\n\t"
                     "buffer_load_dwordx4 %5, %6, %7 offen offset:3072 lds\n\t"
                     "s_mov_b32 m0, %1\n\ts_nop 0\n\t"
                     "buffer_load_dwordx4 %2, %6, %8 offen lds\n\t"
                     "buffer_load_dwordx4 %3, %6, %8 offen offset:1024 lds\n\t"
                     "buffer_load_dwordx4 %4, %6, %8 offen offset:2048 lds\n\t"
                     "buffer_load_dwordx4 %5, %6, %8 offen offset:3072 lds"
                     :: "s"(m), "s"(m + 4096), "v"(wv[0]), "v"(wv[1]), "v"(wv[2]), "v"(wv[3]), "s"(wr), "s"(so), "s"(so + 32 * ND * 2) : "memory");
    };
    unsigned tiles = tile_of(0, cg, a.rank);
#define TILE(r) ((tiles >> (8 * (r))) & 63u)
    unsigned callv;
    asm volatile("buffer_load_dword %0, off, %1, %2 sc1" : "=v"(callv) : "s"(mk_rsrc(myws, WS_NEED)), "s"(CNT_OFF + bid * 64) : "memory");
    if (G == 1) dma(krow(0, w4) + tiles * 128, slot(0, w4));
    unsigned call = 0, parm = 0, exo[3] = {0, 0, 0}, tb[3] = {0, 0, 0}, k0 = 0, m0_ = 0, k1 = 0, m1 = 0, k2 = 0, m2 = 0, oob = 0, fob = 0; int myslot = 0;
    const u32x4 exr = mk_rsrc(myws, ABL == 1 ? 0 : EX_BYTES);
    auto constants = [&]() __attribute__((always_inline)) {
        asm volatile("" : "+v"(callv));
        call = rfl(callv) + 1; const unsigned par = call & 1;
        myslot = PACE ? (tm + (PACE == 2 ? 2 * mi_ : 0)) & 3 : 0;
        parm = par * 0x10001u;
#pragma unroll
        for (int s = 0; s < 3; ++s) {
            const int k = (kq + 1 + s) & 3;
            exo[s] = rfl((unsigned)(bid >> 2) * 65536u + (G == 0 ? slot_off(kq, k, par) + mi_ * 2048 : w4 == kq ? EX_BYTES : slot_off(w4, kq, par)));
            tb[s] = rfl(G == 0 ? slot_bit(kq, k, par) : w4 == kq ? 0u : slot_bit(w4, kq, par));
            PIN_S(exo[s]); PIN_S(tb[s]);
        }
        k0 = krow(G, 4 + w4); m0_ = slot(G, 4 + w4); k1 = krow(G ^ 1, w4); m1 = slot(G ^ 1, w4); k2 = krow(G, 4 + (w4 & 1)); m2 = slot(G, 4 + (w4 & 1));
        oob = (unsigned)((a.rank * MS + tm * 128 + kq * 32 + mi_ * 16) * NS * 2); fob = RF_OFF + (unsigned)((tm * 8 + kq * 2 + mi_) * 128 + a.rank * 4);
        PIN_S(call); PIN_S(parm); PIN_S(myslot); PIN_S(k0); PIN_S(m0_); PIN_S(k1); PIN_S(m1); PIN_S(k2); PIN_S(m2); PIN_S(oob); PIN_S(fob);
    };
    bf16x8 xf[16][2];
    u32x4 tch = {0, 0, 0, 0};
    unsigned wa[4];
    {
        const u32x4 xr = mk_rsrc(a.x, MS * KD * 2);
        unsigned xv[4];
#pragma unroll
        for (int j = 0; j < 4; ++j) { const int m = j * 8 + r8; xv[j] = (unsigned)((tm * 128 + w4 * 32 + m) * KD * 2 + ((c8 ^ ((m >> 1) & 7)) << 4)) - j * 1024; }
        const unsigned xl = lds0 + 65536 + w4 * 8192, xh = lds0 + L_HO + w4 * 4096;
        auto xload = [&](int i, int sl) {
            asm volatile("s_mov_b32 m0, %0\n\ts_nop 0\n\t"
                         "buffer_load_dwordx4 %1, %5, %6 offen lds\n\t"
                         "buffer_load_dwordx4 %2, %5, %6 offen offset:1024 lds\n\t"
                         "buffer_load_dwordx4 %3, %5, %6 offen offset:2048 lds\n\t"
                         "buffer_load_dwordx4 %4, %5, %6 offen offset:3072 lds"
                         :: "s"(sl == 2 ? xh : xl + (sl & 1) * 4096 + (sl < 2 ? 32768 : 0)), "v"(xv[0]), "v"(xv[1]), "v"(xv[2]), "v"(xv[3]), "s"(xr), "s"((unsigned)(kq * 2048 + G * 1024 + ((i + rot) & 7) * 128)) : "memory");
        };
        if (G == 0) { xload(0, 0); xload(1, 1); xload(2, 2); xload(3, 3); xload(4, 4); }
        tiles |= (tile_of(1, cg, a.rank) << 8) | (tile_of(2, cg, a.rank) << 16) | (tile_of(3, cg, a.rank) << 24); PIN_S(tiles);
        { const int q4 = (lane & 15) >> 2, p4 = lane & 3;
#pragma unroll
          for (int ni = 0; ni < 4; ++ni) { const int k = 8 * g4 + q4; const int f = ((k >> 1) & 1) | (((k >> 3) & 1) << 1); wa[ni] = lds0 + L_W + G * 65536 + k * 128 + ((ni ^ f) << 5) + 8 * p4; asm volatile("" : "+v"(wa[ni])); } }
        if (G == 1) {
            if (TOUCH) {
                const unsigned so = krow(1, w4 >> 1) + TILE(0) * 128 + (w4 & 1) * (32 * ND * 2), sx = kq * 2048 + 1024 + (rot & 7) * 128;
#pragma unroll
                for (int j = 0; j < 4; ++j) asm volatile("buffer_load_dwordx4 %0, %1, %2, %3 offen offset:%7\n\tbuffer_load_dwordx4 %0, %4, %5, %6 offen offset:%7" : "+v"(tch)
                                                         : "v"(wv[j]), "s"(wr), "s"(so), "v"(xv[j]), "s"(xr), "s"(sx), "n"(j * 1024) : "memory");
            }
            __builtin_amdgcn_s_setprio(0);
            __builtin_amdgcn_s_sleep(W2DELAY);
            dma(krow(0, 4 + w4) + TILE(0) * 128, slot(0, 4 + w4));
            __builtin_amdgcn_s_setprio(SPRIO);
            waitv<8 + 8 * TOUCH>(); constants(); STAMP(3); BAR;
            xload(0, 0); xload(1, 1); xload(2, 2); xload(3, 3); xload(4, 4);
            STAMP(5);
        }
        unsigned xa[2], xb[2];
#pragma unroll
        for (int ks = 0; ks < 2; ++ks) { const unsigned o = fr * 128 + (((4 * ks + g4) ^ ((fr >> 1) & 7)) << 4); xa[ks] = xl + o; xb[ks] = xh + o; }
#pragma unroll
        for (int i = 0; i < 8; ++i) {
            const int sl = i % 5;
            if (i <= 3) waitv<16>(); else if (i == 4) waitv<12>();
            else if (G == 0) { if (i == 5) waitv<8>(); else if (i == 6) waitv<4>(); else waitv<0>(); }
            else { if (i == 5) waitv<24>(); else if (i == 6) waitv<20>(); else waitv<16>(); }
#pragma unroll
            for (int ks = 0; ks < 2; ++ks)
                asm volatile("ds_read_b128 %0, %2 offset:%3\n\tds_read_b128 %1, %2 offset:%4" : "=&v"(xf[2 * i + ks][0]), "=&v"(xf[2 * i + ks][1]) : "v"(sl == 2 ? xb[ks] : xa[ks]),
                             "n"(sl == 2 ? 0 : (sl & 1) * 4096 + (sl < 2 ? 32768 : 0)), "n"((sl == 2 ? 0 : (sl & 1) * 4096 + (sl < 2 ? 32768 : 0)) + 2048));
            if (i < 3) { waitl<0>(); xload(i + 5, sl); }
            if (i == C24I && G == 1) { BAR; STAMP(6); }
            if (i == 4 && G == 1) { waitl<0>(); dma(krow(1, w4) + TILE(0) * 128, slot(1, w4)); BAR; dma(krow(0, w4) + TILE(1) * 128, slot(0, w4)); }
        }
        waitl<0>();
        STAMP(1);
        if (G == 0) BAR;
        else dma(krow(1, 4 + w4) + TILE(0) * 128, slot(1, 4 + w4));
    }
    if (G == 1) waitv<16>();
    asm volatile("" :: "v"(tch));

    typedef u32x4 IV[3][2];
    auto import_issue = [&](IV& iv, int r, bool wait) __attribute__((always_inline)) {
        const unsigned ro = (unsigned)r << 22;
#define IMPORT_LOADS "buffer_load_dwordx4 %0, %6, %7, %8 offen sc1\n\tbuffer_load_dwordx4 %1, %6, %7, %8 offen offset:1024 sc1\n\t" \
                     "buffer_load_dwordx4 %2, %6, %7, %9 offen sc1\n\tbuffer_load_dwordx4 %3, %6, %7, %9 offen offset:1024 sc1\n\t" \
                     "buffer_load_dwordx4 %4, %6, %7, %10 offen sc1\n\tbuffer_load_dwordx4 %5, %6, %7, %10 offen offset:1024 sc1"
        if (wait) asm volatile(IMPORT_LOADS "\n\ts_waitcnt vmcnt(0)" : "=&v"(iv[0][0]), "=&v"(iv[0][1]), "=&v"(iv[1][0]), "=&v"(iv[1][1]), "=&v"(iv[2][0]), "=&v"(iv[2][1])
                               : "v"(lane * 16), "s"(exr), "s"(exo[0] + ro), "s"(exo[1] + ro), "s"(exo[2] + ro) : "memory");
        else asm volatile(IMPORT_LOADS : "+v"(iv[0][0]), "+v"(iv[0][1]), "+v"(iv[1][0]), "+v"(iv[1][1]), "+v"(iv[2][0]), "+v"(iv[2][1])
                          : "v"(lane * 16), "s"(exr), "s"(exo[0] + ro), "s"(exo[1] + ro), "s"(exo[2] + ro) : "memory");
    };
    int ann = -1;
#define ANN_RSRC(an) mk_rsrc(a.ws[((an) >> 3) & 7], (an) < 0 ? 0 : WS_NEED)
#define ANN_OFF(an) (fob + ((an) & 7) * 4096u)
    auto finish = [&](IV& iv, int r) __attribute__((always_inline)) {
        const u32x4 o[2] = {*(const u32x4*)(smem + L_OWN + mi_ * 2048 + lane * 16), *(const u32x4*)(smem + L_OWN + mi_ * 2048 + 1024 + lane * 16)};
        unsigned nan_ = 0;
        for (unsigned n = 1;; ++n) {
            unsigned miss = 0;
#pragma unroll
            for (int s = 0; s < 3; ++s) miss |= ((iv[s][0][0] ^ parm) | (iv[s][1][0] ^ parm)) & tb[s];
            if (ABL == 1 || __builtin_expect(__all(miss == 0), 1)) break;
            unsigned* const tmo = (unsigned*)(myws + TMO_OFF);
            if (n > RETRY_LIMIT || ((n & 63) == 0 && rfl(__hip_atomic_load(tmo, __ATOMIC_RELAXED, __HIP_MEMORY_SCOPE_AGENT)) == call)) {
                __hip_atomic_store(tmo, call, __ATOMIC_RELAXED, __HIP_MEMORY_SCOPE_AGENT); nan_ = 0x7fc07fc0u; break; }
            import_issue(iv, r, true);
        }
        STAMP(40 + r);
        unsigned char* const ob = smem + L_OUT + mi_ * OUT_B + fr * OUT_RS + g4 * 8;
#pragma unroll
        for (int h = 0; h < 2; ++h) {
            float lo[4], hi[4];
#pragma unroll
            for (int e = 0; e < 4; ++e) { lo[e] = bf_lo(o[h][e]); hi[e] = bf_hi(o[h][e]); }
#pragma unroll
            for (int s = 0; s < 3; ++s) {
                const unsigned w0 = iv[s][h][0] & ~tb[s];
                lo[0] = __builtin_fmaf(bf_lo(w0), tb[s] == 1u ? 1.0027f : 1.0f, lo[0]);
                hi[0] = __builtin_fmaf(bf_hi(w0), tb[s] == 1u ? 1.0f : 1.0027f, hi[0]);
#pragma unroll
                for (int e = 1; e < 4; ++e) { lo[e] += bf_lo(iv[s][h][e]); hi[e] += bf_hi(iv[s][h][e]); }
            }
            u32x2 o0, o1; o0.x = cvt_pk(lo[0], hi[0]) | nan_; o0.y = cvt_pk(lo[1], hi[1]); o1.x = cvt_pk(lo[2], hi[2]); o1.y = cvt_pk(lo[3], hi[3]);
            *(u32x2*)(ob + h * 64) = o0; *(u32x2*)(ob + h * 64 + 32) = o1;
        }
        waitl<0>();
    };
    auto send = [&](int r) __attribute__((always_inline)) {
        const unsigned t = TILE(r); const int an = rfl(ann);
        const u32x4 ro = mk_rsrc(a.out[t >> 3], (ABL == 2 || ABL == 1) && (int)(t >> 3) != a.rank ? 0 : NR * MS * NS * 2);
        const unsigned char* const ob = smem + L_OUT + mi_ * OUT_B + r8 * OUT_RS + c8 * 16;
        const u32x4 v0 = *(const u32x4*)ob, v1 = *(const u32x4*)(ob + 8 * OUT_RS);
        const unsigned oo = oob + (t & 7) * 128;
        asm volatile("s_nop 4\n\tbuffer_store_dwordx4 %0, %2, %3, %4 offen sc0 sc1\n\tbuffer_store_dwordx4 %1, %2, %3, %5 offen sc0 sc1\n\tbuffer_store_dword %6, off, %7, %8 sc0 sc1\n\ts_nop 1"
                     :: "v"(v0), "v"(v1), "v"(r8 * NS * 2 + c8 * 16), "s"(ro), "s"(oo), "s"(oo + 8u * NS * 2), "v"(call), "s"(ANN_RSRC(an)), "s"(ANN_OFF(an)) : "memory");
        ann = t;
        STAMP(44 + r);
    };

    int ys = 0;
#pragma unroll 1
    for (int r = 0; r < 4; ++r) {
        f32x4 acc[2][4];
        IV iv;
        const bool imp = wid < 2 && r > 0;
        {
            bf16x4 wf[8][2];
            auto rdw = [&](int q, int h) __attribute__((always_inline)) {
                if (q < 64) asm volatile("ds_read_b64_tr_b16 %0, %1 offset:%2" : "=v"(wf[q & 7][h]) : "v"(wa[q & 3]), "n"((q >> 3) * 8192 + (((q >> 2) & 1) * 32 + h * 4) * 128));
            };
            auto mm = [&](int q, int mi) __attribute__((always_inline)) {
                const bf16x8 wq = __builtin_shufflevector(wf[q & 7][0], wf[q & 7][1], 0, 1, 2, 3, 4, 5, 6, 7);
                if (q < 4) asm volatile("v_mfma_f32_16x16x32_bf16 %0, %1, %2, 0" : "=&v"(acc[mi][q & 3]) : "v"(wq), "v"(xf[q >> 2][mi]));
                else asm volatile("v_mfma_f32_16x16x32_bf16 %0, %1, %2, %0" : "+v"(acc[mi][q & 3]) : "v"(wq), "v"(xf[q >> 2][mi]));
            };
            STAMP(20 + r);
            waitl<0>();
            if (r == 0 && G == 1) BAR;
#pragma unroll
            for (int q = 0; q < 7; ++q) { rdw(q, 0); rdw(q, 1); }
            if (r > 0) BAR;
            __builtin_amdgcn_s_setprio(0);
            sfor<0, 64>([&](auto Q) __attribute__((always_inline)) {
                constexpr int q = Q;
                if constexpr (!(q & 1)) waitl<(q <= 56 ? 10 : 2 * (62 - q))>();
                if (q == 24) { if (__builtin_expect(r == 0, 0)) { waitv<0>(); BAR; } }
                if (q == MIDA || q == MIDB) { if (MIDA == MIDB || G == (q == MIDB)) { if (__builtin_expect(ys == 0, 1)) waitv<0>(); else waitv<3>(); BAR; } }
                if (q == IMPQ) {
#pragma unroll
                    for (int s = 0; s < 3; ++s) { asm volatile("" : "=v"(iv[s][0]), "=v"(iv[s][1])); }
                    if (__builtin_expect(imp, 0)) { asm volatile("" : "+v"(lane)); import_issue(iv, r - 1, false); }
                }
                mm(q, 0); rdw(q + 7, 0); mm(q, 1); rdw(q + 7, 1);
            });
            __builtin_amdgcn_s_setprio(SPRIO);
            if (G == 0 || r < 3) BAR;
            STAMP(12 + r);
        }
        if (r == 0 && G == 0) constants();
        asm volatile("s_nop 3" : "+v"(lane), "+s"(wid), "+s"(myslot), "+s"(m0_), "+s"(m1), "+s"(m2), "+s"(tb[0]), "+s"(tb[1]), "+s"(tb[2]));
        ys = 0;
        const unsigned cn = TILE(r + 1) * 128;
        if (G == 0) {
            if (!imp && r < 3) { if (r > 0) dma(k2 + cn, m2); dma(k0 + cn, m0_); }
#pragma unroll
            for (int c = 0; c < 4; ++c) { u32x4 pk;
#pragma unroll
                for (int e = 0; e < 4; ++e) { const f32x4 s = acc[c >> 1][2 * (c & 1) + (e >> 1)]; pk[e] = cvt_pk(s[2 * (e & 1)], s[2 * (e & 1) + 1]); }
                *(u32x4*)(smem + L_HO + w4 * 4096 + c * 1024 + lane * 16) = pk; }
            if (r == 0) BAR;
            if (imp) waitv<0>();
            asm volatile("" : "+v"(iv[0][0]), "+v"(iv[0][1]), "+v"(iv[1][0]), "+v"(iv[1][1]), "+v"(iv[2][0]), "+v"(iv[2][1]) :: "memory");
            if (imp) { finish(iv, r - 1); if (myslot == 0) send(r - 1); }
            STAMP(16 + r);
            waitl<0>();
            BAR;
            STAMP(24 + r);
            if (!imp) waitv<0>();
            if (r < 3) dma(k1 + cn, m1);
            if (imp && (myslot == 1 || (r == 3 && myslot > 1))) { for (int i = 1; i < myslot; ++i) __builtin_amdgcn_s_sleep(SDELAY); send(r - 1); ys = 3; }
            if (r == 3) BAR;
        } else {
            u32x4 pk[4];
#pragma unroll
            for (int c = 0; c < 4; ++c) pk[c] = *(const u32x4*)(smem + L_HO + w4 * 4096 + c * 1024 + lane * 16);
            const bool snd = w4 < 2 && r > 0 && r < 3;
            const unsigned eo = exo[0] + ((unsigned)r << 22);
#pragma unroll
            for (int c = 0; c < 4; ++c) {
#pragma unroll
                for (int e = 0; e < 4; ++e) { const f32x4 s = acc[c >> 1][2 * (c & 1) + (e >> 1)]; pk[c][e] = cvt_pk(s[2 * (e & 1)] + bf_lo(pk[c][e]), s[2 * (e & 1) + 1] + bf_hi(pk[c][e])); }
                pk[c][0] = (pk[c][0] & ~tb[0]) | (parm & tb[0]);
                asm volatile("buffer_store_dwordx4 %0, %1, %2, %3 offen offset:%4 sc1" :: "v"(pk[c]), "v"(lane * 16), "s"(exr), "s"(eo), "n"(c * 1024) : "memory");
            }
            STAMP(16 + r);
            if (r < 3) dma(k0 + cn, m0_);
            if (snd && myslot == 2) send(r - 1);
            BAR;
            STAMP(24 + r);
            if (r < 2) dma(k1 + TILE(r + 2) * 128, m1);
            if (tb[0] == 0) {
#pragma unroll
                for (int c = 0; c < 4; ++c) *(u32x4*)(smem + L_OWN + c * 1024 + lane * 16) = pk[c];
            }
            STAMP(32 + r);
            if (snd && myslot == 2) { if (r < 2) waitv<11>(); else waitv<3>(); } else if (r < 2) waitv<8>(); else if (r == 2) waitv<0>();
            if (snd && myslot == 3) { send(r - 1); ys = 3; }
            if (r == 3) { waitl<0>(); BAR; }
        }
    }
    if (G == 0) {
        BAR;
        if (wid < 2) {
            __builtin_amdgcn_s_sleep(RDELAY);
            IV iv;
            import_issue(iv, 3, true);
            finish(iv, 3);
            send(3);
        }
    }
    waitv<0>(); STAMP(48);
    { const int an = rfl(ann); asm volatile("s_nop 4\n\tbuffer_store_dword %0, off, %1, %2 sc0 sc1" :: "v"(call), "s"(ANN_RSRC(an)), "s"(ANN_OFF(an)) : "memory"); }
    if (wid == 7) {
        const unsigned long long t0 = __builtin_amdgcn_s_memrealtime();
        unsigned* const tmo = (unsigned*)(myws + TMO_OFF);
        unsigned* const rf = (unsigned*)(myws + RF_OFF + bid * 128) + (lane & 7);
        bool bad = false;
        for (unsigned n = 1;; ++n) {
            const unsigned f = __hip_atomic_load(rf, __ATOMIC_RELAXED, __HIP_MEMORY_SCOPE_SYSTEM);
            if (__all((lane & 7) == a.rank || f >= call)) break;
            if ((n & 63) == 0 && (__hip_atomic_load(tmo, __ATOMIC_RELAXED, __HIP_MEMORY_SCOPE_AGENT) == call || __builtin_amdgcn_s_memrealtime() - t0 > DEADLINE)) { bad = true; break; }
            __builtin_amdgcn_s_sleep(2);
        }
        if (lane == 0) {
            if (bad) {
                __hip_atomic_store(tmo, call, __ATOMIC_RELAXED, __HIP_MEMORY_SCOPE_AGENT);
                *(unsigned short*)(a.out[a.rank] + (size_t)(a.rank * MS + (bid & 31) * 16) * NS * 2 + (bid >> 5) * 128) = 0x7fc0;
            }
            __hip_atomic_store((unsigned*)(myws + CNT_OFF + bid * 64), call, __ATOMIC_RELAXED, __HIP_MEMORY_SCOPE_AGENT);
        }
        STAMP(49);
    }
    STAMP_OUT();
#undef G
#undef w4
#undef mi_
#undef r8
#undef c8
#undef fr
#undef g4
}
}

extern "C" void kernel_launch(int rank, int nranks, ncclComm_t comm, hipStream_t stream,
                              void* const* d_in, const int* in_sizes, int n_in,
                              void* d_out, int out_size, const int* in_dtypes, int out_dtype,
                              void* const* d_out_all, void* const* d_ws_all, size_t ws_bytes) {
    if (nranks != NR || (unsigned)rank >= (unsigned)NR || ws_bytes < WS_NEED || n_in < 2 || in_sizes[0] != MS * KD || in_sizes[1] != KD * ND || out_size != NR * MS * NS) {
        fprintf(stderr, "kernel_launch: unexpected problem shape\n"); return; }
    Args a;
    a.x = (const unsigned short*)d_in[0]; a.w = (const unsigned short*)d_in[1]; a.rank = rank;
    for (int p = 0; p < NR; ++p) { a.out[p] = (unsigned char*)d_out_all[p]; a.ws[p] = (unsigned char*)d_ws_all[p]; }
    gemm_a2a<<<NWG, 512, 0, stream>>>(a);
    hipError_t e = hipGetLastError();
    if (e != hipSuccess) fprintf(stderr, "HIP error %s\n", hipGetErrorString(e));
}
```

```cpp
#include <hip/hip_runtime.h>
#include <rccl/rccl.h>
#include <cstdio>
#include <cstdint>
#include <type_traits>

#define PRIO2 1
#define SPRIO 3
#define TOUCH 1
#define EARLY 0
#define PACE 2
#define IMPQ 48
#define SDELAY 8
#define TDELAY 20
#define APRIO0 1
#define W2DELAY 24
#define RETRY_LIMIT 600000u
#define MIDQ 32
#define MSL 1
#define HOL2 0
#define MADD 1
#define MFIN 1
#define HOF 1
#define OWNF 1
#define LASTP 0
#define MIDA 38
#define MIDB 40
#define C24I 0
#define HOLATE 1
#define KWARM 1
#define RDELAY 10
#define STAMPS 0
#define ABL 0

namespace {
typedef short bf16x8 __attribute__((ext_vector_type(8)));
typedef short bf16x4 __attribute__((ext_vector_type(4)));
typedef float f32x4 __attribute__((ext_vector_type(4)));
typedef unsigned u32x4 __attribute__((ext_vector_type(4)));
typedef unsigned u32x2 __attribute__((ext_vector_type(2)));
#define LAS __attribute__((address_space(3)))

constexpr int NR = 8, MS = 512, KD = 4096, ND = 4096, NS = 512;
constexpr int NWG = 256;
constexpr unsigned long long DEADLINE = 50000000ull;
constexpr unsigned EX_BYTES = 16u << 20;
constexpr unsigned RF_OFF = EX_BYTES;
constexpr unsigned CNT_OFF = EX_BYTES + 131072;
constexpr unsigned TMO_OFF = EX_BYTES + 196608;
constexpr unsigned DBG_OFF = EX_BYTES + 262144;
constexpr unsigned WS_NEED = EX_BYTES + (1u << 20);
constexpr int OUT_RS = 144, OUT_B = 16 * OUT_RS;
constexpr int L_W = 0;
constexpr int L_HO = 131072, L_OWN = L_HO + 16384, L_OUT = L_OWN + 4096, L_MID = L_OWN + 12288  , L_END = L_OWN + 16384;
static_assert(L_END <= 163840, "lds map");

struct Args {
    const unsigned short* x; const unsigned short* w; unsigned char* myws;
    int rank, pad_[9];
    unsigned char* out[NR]; unsigned char* ws[NR];
};

template <int I, int N, class F> __device__ __forceinline__ void sfor(F&& f) { if constexpr (I < N) { f(std::integral_constant<int, I>{}); sfor<I + 1, N>(f); } }
template <int N> __device__ __forceinline__ void waitv() { asm volatile("s_waitcnt vmcnt(%0)" :: "n"(N) : "memory"); }
template <int N> __device__ __forceinline__ void waitl() { asm volatile("s_waitcnt lgkmcnt(%0)" :: "n"(N) : "memory"); }
__device__ __forceinline__ unsigned cvt_pk(float lo, float hi) { unsigned r; asm("v_cvt_pk_bf16_f32 %0, %1, %2" : "=v"(r) : "v"(lo), "v"(hi)); return r; }
typedef __bf16 bf2_t __attribute__((ext_vector_type(2)));
typedef __bf16 bf8_t __attribute__((ext_vector_type(8)));
__device__ __forceinline__ unsigned cvt2(float lo, float hi) { bf2_t v = {(__bf16)lo, (__bf16)hi}; return __builtin_bit_cast(unsigned, v); }
#define MFMA(a, b, c) __builtin_amdgcn_mfma_f32_16x16x32_bf16(__builtin_bit_cast(bf8_t, a), __builtin_bit_cast(bf8_t, b), c, 0, 0, 0)
__device__ __forceinline__ u32x4 mk_rsrc(const void* p, unsigned bytes) { const unsigned long long a = (unsigned long long)p; u32x4 r; r.x = (unsigned)a; r.y = (unsigned)(a >> 32); r.z = bytes; r.w = 0x00020000; return r; }
__device__ __forceinline__ unsigned lds_rd(unsigned addr) { unsigned v; asm volatile("ds_read_b32 %0, %1\n\ts_waitcnt lgkmcnt(0)" : "=v"(v) : "v"(addr) : "memory"); return v; }
__device__ __forceinline__ void lds_wr(unsigned addr, unsigned v) { asm volatile("ds_write_b32 %0, %1" :: "v"(addr), "v"(v) : "memory"); }
__device__ __forceinline__ unsigned rfl(unsigned v) { return __builtin_amdgcn_readfirstlane(v); }
template <int SL> __device__ __forceinline__ void lds_wait(unsigned addr, unsigned need) { while (__any(lds_rd(addr) < need)) { if (SL) __builtin_amdgcn_s_sleep(SL); } }
__device__ __forceinline__ float bf_lo(unsigned v) { return __uint_as_float(v << 16); }
__device__ __forceinline__ float bf_hi(unsigned v) { return __uint_as_float(v & 0xffff0000u); }
__device__ __forceinline__ int tile_of(int r, int cg, int rank) {
    const int n = r < 3 ? r * 16 + cg : 48 + (cg >> 1), ct = (n * 37) >> 8;
    return r == 3 && (cg & 1) ? rank * 8 + (cg >> 1) : ((rank + 1 + n - 7 * ct) & 7) * 8 + ct;
}
__device__ __forceinline__ unsigned slot_off(int o, int k, unsigned par) { return (unsigned)(par ? k * 4 + o : o * 4 + k) * 4096u; }
__device__ __forceinline__ unsigned slot_bit(int o, int k, unsigned par) { return (par ? k < o : o < k) ? 1u : 0x10000u; }

#define STAMP(i) do {} while (0)
#define STAMP_OUT() do {} while (0)

constexpr int jit_wait(int q) { int n = 0; for (int p = q - 6; p < q; ++p) { if (p > q - 6 && p + 7 < 64) n += 2; if (p >= 0 && (p & 8) && p < 48) n += 1; } const int cap = (q & 8) && q < 48 ? 9 : 11; return n > cap ? cap : n; }
#define BAR asm volatile("s_barrier" ::: "memory")
#define PIN_S(x) asm volatile("" : "+s"(x))
__global__ __launch_bounds__(512, 2) void gemm_a2a(Args a) {
    __shared__ __attribute__((aligned(16))) unsigned char smem[L_END];
    const int tid = threadIdx.x, bid = blockIdx.x;
    int wid = __builtin_amdgcn_readfirstlane(tid >> 6);
    int lane = tid & 63;
    #define G (wid >> 2)
#define w4 (wid & 3)
#define mi_ (wid & 1)
    const int kq = bid & 3, tm = (bid >> 3) & 3, cg = ((bid >> 2) & 1) * 8 + (bid >> 5);
    const int rot = (bid >> 5) + 2 * tm;
    const unsigned lds0 = rfl((unsigned)(size_t)(LAS unsigned char*)smem);
    unsigned char* const myws = a.myws;
    if (KWARM) { unsigned char* p0 = a.out[bid & 7]; unsigned char* p1 = a.ws[bid & 7]; asm volatile("" :: "s"(p0), "s"(p1)); }
#define r8 (lane >> 3)
#define c8 (lane & 7)
#define fr (lane & 15)
#define g4 (lane >> 4)
    unsigned stampv = 0; (void)stampv;
    STAMP(0);
    if (G == 1) __builtin_amdgcn_s_setprio(SPRIO);

    const u32x4 wr = mk_rsrc(a.w, (unsigned)KD * ND * 2);
    unsigned wv[4];
#pragma unroll
    for (int j = 0; j < 4; ++j) { const int f = ((r8 >> 1) & 1) | ((j & 1) << 1); wv[j] = (unsigned)((j * 8 + r8) * ND * 2 + ((c8 ^ (f << 1)) << 4)) - j * 1024; }
    auto krow = [&](int gg, int i) { return (unsigned)(kq * 1024 + gg * 512 + ((i + rot) & 7) * 64) * (ND * 2); };
    auto slot = [&](int gg, int i) { return lds0 + L_W + gg * 65536 + i * 8192; };
    auto dma = [&](unsigned so, unsigned m) __attribute__((always_inline)) {
        asm volatile("s_mov_b32 m0, %0\n\ts_nop 0\n\t"
                     "buffer_load_dwordx4 %2, %6, %7 offen lds\n\t"
                     "buffer_load_dwordx4 %3, %6, %7 offen offset:1024 lds\n\t"
                     "buffer_load_dwordx4 %4, %6, %7 offen offset:2048 lds\n\t"
                     "buffer_load_dwordx4 %5, %6, %7 offen offset:3072 lds\n\t"
                     "s_mov_b32 m0, %1\n\ts_nop 0\n\t"
                     "buffer_load_dwordx4 %2, %6, %8 offen lds\n\t"
                     "buffer_load_dwordx4 %3, %6, %8 offen offset:1024 lds\n\t"
                     "buffer_load_dwordx4 %4, %6, %8 offen offset:2048 lds\n\t"
                     "buffer_load_dwordx4 %5, %6, %8 offen offset:3072 lds"
                     :: "s"(m), "s"(m + 4096), "v"(wv[0]), "v"(wv[1]), "v"(wv[2]), "v"(wv[3]), "s"(wr), "s"(so), "s"(so + 32 * ND * 2) : "memory");
    };
    unsigned tiles = tile_of(0, cg, a.rank);
#define TILE(r) ((tiles >> (8 * (r))) & 63u)
    unsigned callv;
    asm volatile("buffer_load_dword %0, off, %1, %2 sc1" : "=v"(callv) : "s"(mk_rsrc(myws, WS_NEED)), "s"(CNT_OFF + bid * 64) : "memory");
    unsigned call = 0, parm = 0, exo[3] = {0, 0, 0}, tb[3] = {0, 0, 0}, k0 = 0, m0_ = 0, k1 = 0, m1 = 0, k2 = 0, m2 = 0, oob = 0, fob = 0; int myslot = 0;
    const u32x4 exr = mk_rsrc(myws, ABL == 1 ? 0 : EX_BYTES);
    auto constants = [&]() __attribute__((always_inline)) {
        asm volatile("v_readfirstlane_b32 %0, %1\n\ts_nop 3" : "=s"(call) : "v"(callv));
        call += 1; const unsigned par = call & 1;
        myslot = PACE ? (tm + (PACE == 2 ? 2 * mi_ : 0)) & 3 : 0;
        parm = par * 0x10001u;
#pragma unroll
        for (int s = 0; s < 3; ++s) {
            const int k = (kq + 1 + s) & 3;
            exo[s] = rfl((unsigned)(bid >> 2) * 65536u + (G == 0 ? slot_off(kq, k, par) + mi_ * 2048 : w4 == kq ? EX_BYTES : slot_off(w4, kq, par)));
            tb[s] = rfl(G == 0 ? slot_bit(kq, k, par) : w4 == kq ? 0u : slot_bit(w4, kq, par));
            PIN_S(exo[s]); PIN_S(tb[s]);
        }
        k0 = krow(G, 4 + w4); m0_ = slot(G, 4 + w4); k1 = krow(G ^ 1, w4); m1 = slot(G ^ 1, w4); k2 = krow(G, 4 + (w4 & 1)); m2 = slot(G, 4 + (w4 & 1));
        oob = (unsigned)((a.rank * MS + tm * 128 + kq * 32 + mi_ * 16) * NS * 2); fob = RF_OFF + (unsigned)((tm * 8 + kq * 2 + mi_) * 128 + a.rank * 4);
        PIN_S(call); PIN_S(parm); PIN_S(myslot); PIN_S(k0); PIN_S(m0_); PIN_S(k1); PIN_S(m1); PIN_S(k2); PIN_S(m2); PIN_S(oob); PIN_S(fob);
    };
    bf16x8 xf[16][2];
    f32x4 acc[2][4];
    bf16x4 wf[8][2];
    unsigned wa[4];
    auto rdw = [&](int q, int h) __attribute__((always_inline)) {
        asm volatile("ds_read_b64_tr_b16 %0, %1 offset:%2" : "=v"(wf[q & 7][h]) : "v"(wa[q & 3]), "n"((q >> 3) * 8192 + (((q >> 2) & 1) * 32 + h * 4) * 128));
    };
    auto mm = [&](int q, int mi) __attribute__((always_inline)) {
        const bf16x8 wq = __builtin_shufflevector(wf[q & 7][0], wf[q & 7][1], 0, 1, 2, 3, 4, 5, 6, 7);
        if (q < 4) asm volatile("v_mfma_f32_16x16x32_bf16 %0, %1, %2, 0" : "=&v"(acc[mi][q & 3]) : "v"(wq), "v"(xf[q >> 2][mi]));
        else asm volatile("v_mfma_f32_16x16x32_bf16 %0, %1, %2, %0" : "+v"(acc[mi][q & 3]) : "v"(wq), "v"(xf[q >> 2][mi]));
    };
    auto handover = [&]() __attribute__((always_inline)) {
#pragma unroll
        for (int c = 0; c < 4; ++c) { u32x4 pk;
#pragma unroll
            for (int e = 0; e < 4; ++e) { const f32x4 s = acc[c >> 1][2 * (c & 1) + (e >> 1)]; pk[e] = cvt_pk(s[2 * (e & 1)], s[2 * (e & 1) + 1]); }
            *(u32x4*)(smem + L_HO + w4 * 4096 + c * 1024 + lane * 16) = pk; }
    };
    {
        const u32x4 xr = mk_rsrc(a.x, MS * KD * 2);
        unsigned xv[4];
#pragma unroll
        for (int j = 0; j < 4; ++j) { const int m = j * 8 + r8; xv[j] = (unsigned)((tm * 128 + w4 * 32 + m) * KD * 2 + ((c8 ^ ((m >> 1) & 7)) << 4)) - j * 1024; }
        auto xst = [](int c) { return c == 0 || c == 3 ? 98304 : c == 1 ? 65536 : c == 5 ? 0 : c == 6 ? 32768 : L_HO; };
        auto load4 = [&](unsigned m, const unsigned (&v)[4], const u32x4 rs, unsigned so) __attribute__((always_inline)) {
            asm volatile("s_mov_b32 m0, %0\n\ts_nop 0\n\t"
                         "buffer_load_dwordx4 %1, %5, %6 offen lds\n\tbuffer_load_dwordx4 %2, %5, %6 offen offset:1024 lds\n\t"
                         "buffer_load_dwordx4 %3, %5, %6 offen offset:2048 lds\n\tbuffer_load_dwordx4 %4, %5, %6 offen offset:3072 lds"
                         :: "s"(m), "v"(v[0]), "v"(v[1]), "v"(v[2]), "v"(v[3]), "s"(rs), "s"(so) : "memory");
        };
        auto chunk_x = [&](int c, int st) __attribute__((always_inline)) { load4(lds0 + xst(c) + w4 * 8192 + st * 4096, xv, xr, kq * 2048 + (c >> 2) * 1024 + ((2 * (c & 3) + st + rot) & 7) * 128); };
        auto chunk_w = [&](int c) __attribute__((always_inline)) { load4(slot(c >> 2, 2 * (c & 3)) + w4 * 4096, wv, wr, krow(c >> 2, 2 * (c & 3) + (w4 >> 1)) + (w4 & 1) * (32 * ND * 2) + (tiles & 63) * 128); };
        auto chunk = [&](int c) __attribute__((always_inline)) { chunk_x(c, 0); chunk_w(c); chunk_x(c, 1); };
        STAMP(4);
        if (G == 1) chunk(0);
        STAMP(5);
        tiles |= (tile_of(1, cg, a.rank) << 8) | (tile_of(2, cg, a.rank) << 16) | (tile_of(3, cg, a.rank) << 24); PIN_S(tiles);
        { const int q4 = (lane & 15) >> 2, p4 = lane & 3;
#pragma unroll
          for (int ni = 0; ni < 4; ++ni) { const int k = 8 * g4 + q4; const int f = ((k >> 1) & 1) | (((k >> 3) & 1) << 1); wa[ni] = lds0 + L_W + G * 65536 + k * 128 + ((ni ^ f) << 5) + 8 * p4; asm volatile("" : "+v"(wa[ni])); } }
        unsigned xo[2];
#pragma unroll
        for (int ks = 0; ks < 2; ++ks) xo[ks] = lds0 + w4 * 8192 + fr * 128 + (((4 * ks + g4) ^ ((fr >> 1) & 7)) << 4);
        if (G == 0) chunk(1);
        waitv<12>(); constants();
        if (G == 1) {
            waitv<0>(); STAMP(2); BAR; chunk(2);
            BAR; chunk(3);
            waitv<12>(); BAR; chunk_w(4); chunk_w(5);
            waitv<8>(); BAR; chunk_x(4, 0); chunk_x(4, 1); chunk_x(5, 0); chunk_x(5, 1);
            waitv<8>(); STAMP(3);
        }
        unsigned xa[2];
        auto xbase = [&](int cc) __attribute__((always_inline)) { unsigned xs = G == 0 ? xst(cc) : xst(4 + cc); PIN_S(xs); xa[0] = xo[0] + xs; xa[1] = xo[1] + xs; };
        auto rdx = [&](int cc, int j) __attribute__((always_inline)) {
            asm volatile("ds_read_b128 %0, %1 offset:%2" : "=v"(xf[4 * cc + (j >> 1)][j & 1]) : "v"(xa[(j >> 1) & 1]), "n"((j >> 2) * 4096 + (j & 1) * 2048));
        };
        xbase(0);
        BAR;
        __builtin_amdgcn_s_setprio(0);
#pragma unroll
        for (int j = 0; j < 8; ++j) rdx(0, j);
        waitl<6>();
#pragma unroll
        for (int q = 0; q < 4; ++q) { rdw(q, 0); rdw(q, 1); }
        waitl<8>();
#pragma unroll
        for (int q = 4; q < 7; ++q) { rdw(q, 0); rdw(q, 1); }
        sfor<0, 64>([&](auto Q) __attribute__((always_inline)) {
            constexpr int q = Q;
            if constexpr (!(q & 1)) waitl<jit_wait(q)>();
            if constexpr ((q & 15) == 8) {
                if (q == 8) waitv<0>();
                if (q < 56) { xbase((q >> 4) + 1); BAR; } else if (G == 1) BAR;
            }
            mm(q, 0); if (q + 7 < 64) rdw(q + 7, 0); mm(q, 1); if (q + 7 < 64) rdw(q + 7, 1);
            if constexpr ((q & 8) && q < 48) rdx((q >> 4) + 1, q & 7);
        });
        __builtin_amdgcn_s_setprio(SPRIO);
        STAMP(12);
        if (G == 0) {
            const unsigned c1 = TILE(1) * 128;
            BAR; chunk(6); chunk_w(7);
            BAR; chunk_x(7, 0); chunk_x(7, 1);
            waitv<12>(); BAR; dma(krow(0, w4) + c1, slot(0, w4));
            waitv<8>(); BAR; dma(k0 + c1, m0_);
            waitv<8>(); BAR; handover();
            lds_wr(lds0 + L_MID + w4 * 512 + lane * 4, 0u); lds_wr(lds0 + L_MID + w4 * 512 + 256 + lane * 4, 0u);
            dma(k1 + c1, m1);
            STAMP(16);
        } else BAR;
    }

    typedef u32x4 IV[3][2];
    auto import_issue = [&](IV& iv, int r, bool wait) __attribute__((always_inline)) {
        const unsigned ro = (unsigned)r << 22;
#define IMPORT_LOADS "buffer_load_dwordx4 %0, %6, %7, %8 offen sc1\n\tbuffer_load_dwordx4 %1, %6, %7, %8 offen offset:1024 sc1\n\t" \
                     "buffer_load_dwordx4 %2, %6, %7, %9 offen sc1\n\tbuffer_load_dwordx4 %3, %6, %7, %9 offen offset:1024 sc1\n\t" \
                     "buffer_load_dwordx4 %4, %6, %7, %10 offen sc1\n\tbuffer_load_dwordx4 %5, %6, %7, %10 offen offset:1024 sc1"
        if (wait) asm volatile(IMPORT_LOADS "\n\ts_waitcnt vmcnt(0)" : "=&v"(iv[0][0]), "=&v"(iv[0][1]), "=&v"(iv[1][0]), "=&v"(iv[1][1]), "=&v"(iv[2][0]), "=&v"(iv[2][1])
                               : "v"(lane * 16), "s"(exr), "s"(exo[0] + ro), "s"(exo[1] + ro), "s"(exo[2] + ro) : "memory");
        else asm volatile(IMPORT_LOADS : "+v"(iv[0][0]), "+v"(iv[0][1]), "+v"(iv[1][0]), "+v"(iv[1][1]), "+v"(iv[2][0]), "+v"(iv[2][1])
                          : "v"(lane * 16), "s"(exr), "s"(exo[0] + ro), "s"(exo[1] + ro), "s"(exo[2] + ro) : "memory");
    };
    int ann = -1;
#define ANN_RSRC(an) mk_rsrc(a.ws[((an) >> 3) & 7], (an) < 0 ? 0 : WS_NEED)
#define ANN_OFF(an) (fob + ((an) & 7) * 4096u)
    auto finish = [&](IV& iv, int r) __attribute__((always_inline)) {
        const u32x4 o[2] = {*(const u32x4*)(smem + L_OWN + mi_ * 2048 + lane * 16), *(const u32x4*)(smem + L_OWN + mi_ * 2048 + 1024 + lane * 16)};
        unsigned nan_ = 0;
        for (unsigned n = 1;; ++n) {
            unsigned miss = 0;
#pragma unroll
            for (int s = 0; s < 3; ++s) miss |= ((iv[s][0][0] ^ parm) | (iv[s][1][0] ^ parm)) & tb[s];
            if (ABL == 1 || __builtin_expect(__all(miss == 0), 1)) break;
            unsigned* const tmo = (unsigned*)(myws + TMO_OFF);
            if (n > RETRY_LIMIT || ((n & 63) == 0 && rfl(__hip_atomic_load(tmo, __ATOMIC_RELAXED, __HIP_MEMORY_SCOPE_AGENT)) == call)) {
                __hip_atomic_store(tmo, call, __ATOMIC_RELAXED, __HIP_MEMORY_SCOPE_AGENT); nan_ = 0x7fc07fc0u; break; }
            import_issue(iv, r, true);
        }
        STAMP(40 + r);
        unsigned char* const ob = smem + L_OUT + mi_ * OUT_B + fr * OUT_RS + g4 * 8;
        {
            u32x4 sel[2];
            { const unsigned one = g4 == (fr >> 2) ? 0x3f80u << (16 * (lane & 1)) : 0u, a0 = (lane & 2) ? 0u : one, a1 = (lane & 2) ? one : 0u;
              sel[0] = (u32x4){a0, a1, 0u, 0u}; sel[1] = (u32x4){0u, 0u, a0, a1}; }
#pragma unroll
            for (int s = 0; s < 3; ++s) { iv[s][0][0] &= ~tb[s]; iv[s][1][0] &= ~tb[s]; }
            f32x4 d[2][2] = {};
#pragma unroll
            for (int s = 0; s < 4; ++s)
#pragma unroll
                for (int h = 0; h < 2; ++h)
#pragma unroll
                    for (int j = 0; j < 2; ++j) d[h][j] = MFMA(sel[j], s < 3 ? iv[s < 3 ? s : 0][h] : o[h], d[h][j]);
#pragma unroll
            for (int h = 0; h < 2; ++h) {
                u32x2 o0, o1; o0.x = cvt2(d[h][0][0], d[h][0][1]) | nan_; o0.y = cvt2(d[h][0][2], d[h][0][3]); o1.x = cvt2(d[h][1][0], d[h][1][1]); o1.y = cvt2(d[h][1][2], d[h][1][3]);
                *(u32x2*)(ob + h * 64) = o0; *(u32x2*)(ob + h * 64 + 32) = o1;
            }
        }
        waitl<0>();
    };
    auto send = [&](int r) __attribute__((always_inline)) {
        const unsigned t = TILE(r); const int an = rfl(ann);
        const u32x4 ro = mk_rsrc(a.out[t >> 3], (ABL == 2 || ABL == 1) && (int)(t >> 3) != a.rank ? 0 : NR * MS * NS * 2);
        const unsigned char* const ob = smem + L_OUT + mi_ * OUT_B + r8 * OUT_RS + c8 * 16;
        const u32x4 v0 = *(const u32x4*)ob, v1 = *(const u32x4*)(ob + 8 * OUT_RS);
        const unsigned oo = oob + (t & 7) * 128;
        asm volatile("s_nop 4\n\tbuffer_store_dwordx4 %0, %2, %3, %4 offen sc0 sc1\n\tbuffer_store_dwordx4 %1, %2, %3, %5 offen sc0 sc1\n\tbuffer_store_dword %6, off, %7, %8 sc0 sc1\n\ts_nop 1"
                     :: "v"(v0), "v"(v1), "v"(r8 * NS * 2 + c8 * 16), "s"(ro), "s"(oo), "s"(oo + 8u * NS * 2), "v"(call), "s"(ANN_RSRC(an)), "s"(ANN_OFF(an)) : "memory");
        ann = t;
        STAMP(44 + r);
    };

    int ys = 0;
    IV iv;
#pragma unroll
    for (int s = 0; s < 3; ++s) { asm volatile("" : "=v"(iv[s][0]), "=v"(iv[s][1])); }
#pragma unroll 1
    for (int r = 0;; ++r) {
        asm volatile("s_nop 3" : "+v"(lane), "+s"(wid), "+s"(myslot), "+s"(m0_), "+s"(m1), "+s"(m2), "+s"(tb[0]), "+s"(tb[1]), "+s"(tb[2]));
        ys = 0;
        if (LASTP && r == 3) myslot = (int)(TILE(3) >> 3) == a.rank ? LASTP + (myslot & 1) : (LASTP == 3 ? 0 : myslot & 1);
        const unsigned cn = TILE(r + 1) * 128;
        if (G == 0) {
            if (r > 0) {
                const bool imp = wid < 2;
                if (!HOF && !imp && r < 3) { dma(k2 + cn, m2); dma(k0 + cn, m0_); }
                if (!(HOL2 && imp)) handover();
                if (HOF && !imp && r < 3) { waitl<0>(); if (HOF > 1) __builtin_amdgcn_s_sleep(HOF); dma(k2 + cn, m2); dma(k0 + cn, m0_); }
                STAMP(28 + r);
                if (imp) waitv<0>();
                asm volatile("" : "+v"(iv[0][0]), "+v"(iv[0][1]), "+v"(iv[1][0]), "+v"(iv[1][1]), "+v"(iv[2][0]), "+v"(iv[2][1]) :: "memory");
                if (imp) { finish(iv, r - 1); if (myslot == 0) { send(r - 1); ys = 3; } }
                if (HOL2 && imp) handover();
                STAMP(16 + r);
                if (r < 3) lds_wait<MSL>(lds0 + L_MID + (4 + (lane & 3)) * 256, (unsigned)r);
                STAMP(24 + r);
                if (!imp && r < 3) { dma(krow(1, w4 & 1) + cn, slot(1, w4 & 1)); dma(k1 + cn, m1); waitv<16>(); }
                if (r == 3) { waitl<0>(); BAR; }
                if (imp && (myslot == 1 || (r == 3 && myslot > 1))) { for (int i = 1; i < myslot; ++i) __builtin_amdgcn_s_sleep(SDELAY); send(r - 1); ys = 3; }
            }
        } else {
            if (r == 3) BAR;
            u32x4 pk[4];
#pragma unroll
            for (int c = 0; c < 4; ++c) pk[c] = *(const u32x4*)(smem + L_HO + w4 * 4096 + c * 1024 + lane * 16);
            const bool snd = w4 < 2 && r > 0 && r < 3;
            const unsigned eo = exo[0] + ((unsigned)r << 22);
            u32x4 sel[2];
            { const unsigned one = g4 == (fr >> 2) ? 0x3f80u << (16 * (lane & 1)) : 0u, a0 = (lane & 2) ? 0u : one, a1 = (lane & 2) ? one : 0u;
              sel[0] = (u32x4){a0, a1, 0u, 0u}; sel[1] = (u32x4){0u, 0u, a0, a1}; }
            f32x4 sum[4][2];
#pragma unroll
            for (int c = 0; c < 4; ++c)
#pragma unroll
                for (int h = 0; h < 2; ++h) sum[c][h] = MFMA(sel[h], pk[c], acc[c >> 1][2 * (c & 1) + h]);
#pragma unroll
            for (int c = 0; c < 4; ++c) {
#pragma unroll
                for (int e = 0; e < 4; ++e) {
                    const f32x4 s = sum[c][e >> 1];
                    pk[c][e] = e == 0 ? cvt2(s[0] * (tb[0] == 1u ? 1.0027f : 1.0f), s[1] * (tb[0] == 0x10000u ? 1.0027f : 1.0f)) : cvt2(s[2 * (e & 1)], s[2 * (e & 1) + 1]);
                }
                pk[c][0] = (pk[c][0] & ~tb[0]) | (parm & tb[0]);
                asm volatile("buffer_store_dwordx4 %0, %1, %2, %3 offen offset:%4 sc1" :: "v"(pk[c]), "v"(lane * 16), "s"(exr), "s"(eo), "n"(c * 1024) : "memory");
            }
            if (OWNF && tb[0] == 0) {
#pragma unroll
                for (int c = 0; c < 4; ++c) *(u32x4*)(smem + L_OWN + c * 1024 + lane * 16) = pk[c];
            }
            STAMP(16 + r);
            if (r == 0) BAR;
            if (r < 3) dma(k0 + cn, m0_);
            if (snd && myslot == 2) send(r - 1);
            if (r < 3) lds_wait<MSL>(lds0 + L_MID + (lane & 3) * 256, (unsigned)r + 1);
            STAMP(24 + r);
            if (r < 2) dma(k1 + TILE(r + 2) * 128, m1);
            if (!OWNF && tb[0] == 0) {
#pragma unroll
                for (int c = 0; c < 4; ++c) *(u32x4*)(smem + L_OWN + c * 1024 + lane * 16) = pk[c];
            }
            STAMP(32 + r);
            if (snd && myslot == 2) { if (r < 2) waitv<11>(); else waitv<3>(); } else if (r < 2) waitv<8>(); else if (r == 2) waitv<0>();
            if (snd && myslot == 3) { send(r - 1); ys = 3; }
            if (r == 3) { waitl<0>(); BAR; }
        }
        if (r == 3) break;
        STAMP(21 + r);
        unsigned mida = lds0 + L_MID + wid * 256 + lane * 4, midv = r + 1; asm volatile("" : "+v"(mida), "+v"(midv));
        waitl<0>();
#pragma unroll
        for (int q = 0; q < 7; ++q) { rdw(q, 0); rdw(q, 1); }
        BAR;
        __builtin_amdgcn_s_setprio(0);
        sfor<0, 64>([&](auto Q) __attribute__((always_inline)) {
            constexpr int q = Q;
            if constexpr (!(q & 1)) waitl<(q <= 56 ? 10 : 2 * (62 - q))>();
            if (q == 24) { if (__builtin_expect(r == 0 && G == 0, 0)) { waitv<8>(); BAR; } }
            if (q == MIDQ) { if (__builtin_expect(ys == 0, 1)) waitv<0>(); else waitv<3>(); lds_wr(mida, midv); }
            if (q == IMPQ) {
#pragma unroll
                for (int s = 0; s < 3; ++s) { asm volatile("" : "=v"(iv[s][0]), "=v"(iv[s][1])); }
                if (__builtin_expect(wid < 2, 0)) { asm volatile("" : "+v"(lane)); import_issue(iv, r, false); }
            }
            mm(q, 0); if (q + 7 < 64) rdw(q + 7, 0); mm(q, 1); if (q + 7 < 64) rdw(q + 7, 1);
        });
        __builtin_amdgcn_s_setprio(SPRIO);
        if (G == 0 || r < 2) BAR;
        STAMP(13 + r);
    }
    if (G == 0) {
        BAR;
        if (wid < 2) {
            __builtin_amdgcn_s_sleep(RDELAY);
            import_issue(iv, 3, true);
            finish(iv, 3);
            send(3);
        }
    }
    waitv<0>(); STAMP(48);
    { const int an = rfl(ann); asm volatile("s_nop 4\n\tbuffer_store_dword %0, off, %1, %2 sc0 sc1" :: "v"(call), "s"(ANN_RSRC(an)), "s"(ANN_OFF(an)) : "memory"); }
    if (wid == 7) {
        const unsigned long long t0 = __builtin_amdgcn_s_memrealtime();
        unsigned* const tmo = (unsigned*)(myws + TMO_OFF);
        unsigned* const rf = (unsigned*)(myws + RF_OFF + bid * 128) + (lane & 7);
        bool bad = false;
        for (unsigned n = 1;; ++n) {
            const unsigned f = __hip_atomic_load(rf, __ATOMIC_RELAXED, __HIP_MEMORY_SCOPE_SYSTEM);
            if (__all((lane & 7) == a.rank || f >= call)) break;
            if ((n & 63) == 0 && (__hip_atomic_load(tmo, __ATOMIC_RELAXED, __HIP_MEMORY_SCOPE_AGENT) == call || __builtin_amdgcn_s_memrealtime() - t0 > DEADLINE)) { bad = true; break; }
            __builtin_amdgcn_s_sleep(2);
        }
        if (lane == 0) {
            if (bad) {
                __hip_atomic_store(tmo, call, __ATOMIC_RELAXED, __HIP_MEMORY_SCOPE_AGENT);
                *(unsigned short*)(a.out[a.rank] + (size_t)(a.rank * MS + (bid & 31) * 16) * NS * 2 + (bid >> 5) * 128) = 0x7fc0;
            }
            __hip_atomic_store((unsigned*)(myws + CNT_OFF + bid * 64), call, __ATOMIC_RELAXED, __HIP_MEMORY_SCOPE_AGENT);
        }
        STAMP(49);
    }
    STAMP_OUT();
#undef G
#undef w4
#undef mi_
#undef r8
#undef c8
#undef fr
#undef g4
}
}

extern "C" void kernel_launch(int rank, int nranks, ncclComm_t comm, hipStream_t stream,
                              void* const* d_in, const int* in_sizes, int n_in,
                              void* d_out, int out_size, const int* in_dtypes, int out_dtype,
                              void* const* d_out_all, void* const* d_ws_all, size_t ws_bytes) {
    if (nranks != NR || (unsigned)rank >= (unsigned)NR || ws_bytes < WS_NEED || n_in < 2 || in_sizes[0] != MS * KD || in_sizes[1] != KD * ND || out_size != NR * MS * NS) {
        fprintf(stderr, "kernel_launch: unexpected problem shape\n"); return; }
    Args a = {};
    a.x = (const unsigned short*)d_in[0]; a.w = (const unsigned short*)d_in[1]; a.rank = rank; a.myws = (unsigned char*)d_ws_all[rank];
    for (int p = 0; p < NR; ++p) { a.out[p] = (unsigned char*)d_out_all[p]; a.ws[p] = (unsigned char*)d_ws_all[p]; }
    gemm_a2a<<<NWG, 512, 0, stream>>>(a);
    hipError_t e = hipGetLastError();
    if (e != hipSuccess) fprintf(stderr, "HIP error %s\n", hipGetErrorString(e));
}
```

```cpp
#include <hip/hip_runtime.h>
#include <rccl/rccl.h>
#include <cstdio>
#include <cstdint>
#include <type_traits>

namespace {
typedef short bf16x8 __attribute__((ext_vector_type(8)));
typedef short bf16x4 __attribute__((ext_vector_type(4)));
typedef float f32x4 __attribute__((ext_vector_type(4)));
typedef unsigned u32x4 __attribute__((ext_vector_type(4)));
typedef unsigned u32x2 __attribute__((ext_vector_type(2)));
typedef __bf16 bf2_t __attribute__((ext_vector_type(2)));
typedef __bf16 bf8_t __attribute__((ext_vector_type(8)));
#define LAS __attribute__((address_space(3)))

constexpr int NR = 8, MS = 512, KD = 4096, ND = 4096, NS = 512;
constexpr int NWG = 256;
constexpr unsigned long long DEADLINE = 50000000ull;
constexpr int SPRIO = 3;
constexpr int MIDQ = 32, IMPQ = 56;
constexpr int SDELAY = 8, RDELAY = 10;
constexpr unsigned EX_BYTES = 16u << 20;
constexpr unsigned RF_OFF = EX_BYTES;
constexpr unsigned CNT_OFF = EX_BYTES + 131072;
constexpr unsigned TMO_OFF = EX_BYTES + 196608;
constexpr unsigned WS_NEED = EX_BYTES + 262144;
constexpr int OUT_RS = 144, OUT_B = 16 * OUT_RS;
constexpr int L_W = 0, L_HO = 131072, L_OWN = L_HO + 16384, L_OUT = L_OWN + 4096, L_MID = L_OWN + 12288, L_END = L_OWN + 16384;
static_assert(L_OUT + 2 * OUT_B <= L_MID && L_MID + 2048 <= L_END && L_END <= 163840, "lds map");

struct Args {
    const unsigned short* x; const unsigned short* w; unsigned char* myws;
    int rank, pad_[9];
    unsigned char* out[NR]; unsigned char* ws[NR];
};

template <int I, int N, class F> __device__ __forceinline__ void sfor(F&& f) { if constexpr (I < N) { f(std::integral_constant<int, I>{}); sfor<I + 1, N>(f); } }
template <int N> __device__ __forceinline__ void waitv() { asm volatile("s_waitcnt vmcnt(%0)" :: "n"(N) : "memory"); }
template <int N> __device__ __forceinline__ void waitl() { asm volatile("s_waitcnt lgkmcnt(%0)" :: "n"(N) : "memory"); }
__device__ __forceinline__ unsigned cvt2(float lo, float hi) { bf2_t v = {(__bf16)lo, (__bf16)hi}; return __builtin_bit_cast(unsigned, v); }
#define MFMA(a, b, c) __builtin_amdgcn_mfma_f32_16x16x32_bf16(__builtin_bit_cast(bf8_t, a), __builtin_bit_cast(bf8_t, b), c, 0, 0, 0)
__device__ __forceinline__ u32x4 mk_rsrc(const void* p, unsigned bytes) { const unsigned long long a = (unsigned long long)p; u32x4 r; r.x = (unsigned)a; r.y = (unsigned)(a >> 32); r.z = bytes; r.w = 0x00020000; return r; }
__device__ __forceinline__ unsigned lds_rd(unsigned addr) { unsigned v; asm volatile("ds_read_b32 %0, %1\n\ts_waitcnt lgkmcnt(0)" : "=v"(v) : "v"(addr) : "memory"); return v; }
__device__ __forceinline__ void lds_wr(unsigned addr, unsigned v) { asm volatile("ds_write_b32 %0, %1" :: "v"(addr), "v"(v) : "memory"); }
__device__ __forceinline__ void lds_wait(unsigned addr, unsigned need) { while (__any(lds_rd(addr) < need)) __builtin_amdgcn_s_sleep(1); }
__device__ __forceinline__ unsigned rfl(unsigned v) { return __builtin_amdgcn_readfirstlane(v); }
__device__ __forceinline__ unsigned tile_of(int r, int cg, int rank) { const int d = (rank + 1 + cg + 3 * r) & 7; return (unsigned)(d * 8 + ((2 * r + (cg >> 3) + d) & 7)); }
__device__ __forceinline__ unsigned slot_off(int o, int k, unsigned par) { return (unsigned)(par ? k * 4 + o : o * 4 + k) * 4096u; }
__device__ __forceinline__ unsigned slot_bit(int o, int k, unsigned par) { return (par ? k < o : o < k) ? 1u : 0x10000u; }
constexpr int jit_wait(int q) { int n = 0; for (int p = q - 6; p < q; ++p) { if (p > q - 6 && p + 7 < 64) n += 2; if (p >= 0 && (p & 8) && p < 48) n += 1; } const int cap = (q & 8) && q < 48 ? 9 : 11; return n > cap ? cap : n; }
#define BAR asm volatile("s_barrier" ::: "memory")
#define PIN_S(x) asm volatile("" : "+s"(x))

__global__ __launch_bounds__(512, 2) void gemm_a2a(Args a) {
    __shared__ __attribute__((aligned(16))) unsigned char smem[L_END];
    const int tid = threadIdx.x, bid = blockIdx.x;
    int wid = __builtin_amdgcn_readfirstlane(tid >> 6), lane = tid & 63;
#define G (wid >> 2)
#define w4 (wid & 3)
#define mi_ (wid & 1)
#define r8 (lane >> 3)
#define c8 (lane & 7)
#define fr (lane & 15)
#define g4 (lane >> 4)
    const int kq = bid & 3, tm = (bid >> 3) & 3, cg = ((bid >> 2) & 1) * 8 + (bid >> 5);
    const int rot = (bid >> 5) + 2 * tm;
    const unsigned lds0 = rfl((unsigned)(size_t)(LAS unsigned char*)smem);
    unsigned char* const myws = a.myws;
    { unsigned char* p0 = a.out[bid & 7]; unsigned char* p1 = a.ws[bid & 7]; asm volatile("" :: "s"(p0), "s"(p1)); }
    if (G == 1) __builtin_amdgcn_s_setprio(SPRIO);

    const u32x4 wr = mk_rsrc(a.w, (unsigned)KD * ND * 2);
    unsigned wv[4];
#pragma unroll
    for (int j = 0; j < 4; ++j) { const int f = ((r8 >> 1) & 1) | ((j & 1) << 1); wv[j] = (unsigned)((j * 8 + r8) * ND * 2 + ((c8 ^ (f << 1)) << 4)) - j * 1024; }
    auto krow = [&](int gg, int i) { return (unsigned)(kq * 1024 + gg * 512 + ((i + rot) & 7) * 64) * (ND * 2); };
    auto slot = [&](int gg, int i) { return lds0 + L_W + gg * 65536 + i * 8192; };
    auto load4 = [&](unsigned m, const unsigned (&v)[4], const u32x4 rs, unsigned so) __attribute__((always_inline)) {
        asm volatile("s_mov_b32 m0, %0\n\ts_nop 0\n\t"
                     "buffer_load_dwordx4 %1, %5, %6 offen lds\n\tbuffer_load_dwordx4 %2, %5, %6 offen offset:1024 lds\n\t"
                     "buffer_load_dwordx4 %3, %5, %6 offen offset:2048 lds\n\tbuffer_load_dwordx4 %4, %5, %6 offen offset:3072 lds"
                     :: "s"(m), "v"(v[0]), "v"(v[1]), "v"(v[2]), "v"(v[3]), "s"(rs), "s"(so) : "memory");
    };
    auto dma = [&](unsigned so, unsigned m) __attribute__((always_inline)) { load4(m, wv, wr, so); load4(m + 4096, wv, wr, so + 32 * ND * 2); };
    unsigned tiles = tile_of(0, cg, a.rank);
#define TILE(r) ((tiles >> (8 * (r))) & 63u)
    unsigned callv;
    asm volatile("buffer_load_dword %0, off, %1, %2 sc1" : "=v"(callv) : "s"(mk_rsrc(myws, WS_NEED)), "s"(CNT_OFF + bid * 64) : "memory");
    const u32x4 exr = mk_rsrc(myws, EX_BYTES);

    bf16x8 xf[16][2];
    f32x4 acc[2][4];
    bf16x4 wf[8][2];
    unsigned wa[4];
    auto rdw = [&](int q, int h) __attribute__((always_inline)) {
        asm volatile("ds_read_b64_tr_b16 %0, %1 offset:%2" : "=v"(wf[q & 7][h]) : "v"(wa[q & 3]), "n"((q >> 3) * 8192 + (((q >> 2) & 1) * 32 + h * 4) * 128));
    };
    auto mm = [&](int q, int mi) __attribute__((always_inline)) {
        const bf16x8 wq = __builtin_shufflevector(wf[q & 7][0], wf[q & 7][1], 0, 1, 2, 3, 4, 5, 6, 7);
        if (q < 4) asm volatile("v_mfma_f32_16x16x32_bf16 %0, %1, %2, 0" : "=&v"(acc[mi][q & 3]) : "v"(wq), "v"(xf[q >> 2][mi]));
        else asm volatile("v_mfma_f32_16x16x32_bf16 %0, %1, %2, %0" : "+v"(acc[mi][q & 3]) : "v"(wq), "v"(xf[q >> 2][mi]));
    };
    auto settle = [&]() __attribute__((always_inline)) {
        asm volatile("s_nop 7" : "+v"(acc[0][0]), "+v"(acc[0][1]), "+v"(acc[0][2]), "+v"(acc[0][3]), "+v"(acc[1][0]), "+v"(acc[1][1]), "+v"(acc[1][2]), "+v"(acc[1][3]));
    };
    auto handover = [&]() __attribute__((always_inline)) {
#pragma unroll
        for (int c = 0; c < 4; ++c) { u32x4 pk;
#pragma unroll
            for (int e = 0; e < 4; ++e) { const f32x4 s = acc[c >> 1][2 * (c & 1) + (e >> 1)]; pk[e] = cvt2(s[2 * (e & 1)], s[2 * (e & 1) + 1]); }
            *(u32x4*)(smem + L_HO + w4 * 4096 + c * 1024 + lane * 16) = pk; }
    };
    unsigned call, parm, exo[3], tb[3], k0, m0_, k1, m1, k2, m2, oob, fob; int myslot;
    {
        const u32x4 xr = mk_rsrc(a.x, MS * KD * 2);
        unsigned xv[4];
#pragma unroll
        for (int j = 0; j < 4; ++j) { const int m = j * 8 + r8; xv[j] = (unsigned)((tm * 128 + w4 * 32 + m) * KD * 2 + ((c8 ^ ((m >> 1) & 7)) << 4)) - j * 1024; }
        auto xst = [](int c) { return c == 0 || c == 3 ? 98304 : c == 1 ? 65536 : c == 5 ? 0 : c == 6 ? 32768 : L_HO; };
        auto chunk_x = [&](int c, int st) __attribute__((always_inline)) { load4(lds0 + xst(c) + w4 * 8192 + st * 4096, xv, xr, kq * 2048 + (c >> 2) * 1024 + ((2 * (c & 3) + st + rot) & 7) * 128); };
        auto chunk_w = [&](int c) __attribute__((always_inline)) { load4(slot(c >> 2, 2 * (c & 3)) + w4 * 4096, wv, wr, krow(c >> 2, 2 * (c & 3) + (w4 >> 1)) + (w4 & 1) * (32 * ND * 2) + (tiles & 63) * 128); };
        auto chunk = [&](int c) __attribute__((always_inline)) { chunk_x(c, 0); chunk_w(c); chunk_x(c, 1); };
        if (G == 1) chunk(0);
        tiles |= (tile_of(1, cg, a.rank) << 8) | (tile_of(2, cg, a.rank) << 16) | (tile_of(3, cg, a.rank) << 24); PIN_S(tiles);
        { const int q4 = (lane & 15) >> 2, p4 = lane & 3;
#pragma unroll
          for (int ni = 0; ni < 4; ++ni) { const int k = 8 * g4 + q4; const int f = ((k >> 1) & 1) | (((k >> 3) & 1) << 1); wa[ni] = lds0 + L_W + G * 65536 + k * 128 + ((ni ^ f) << 5) + 8 * p4; asm volatile("" : "+v"(wa[ni])); } }
        unsigned xo[2];
#pragma unroll
        for (int ks = 0; ks < 2; ++ks) xo[ks] = lds0 + w4 * 8192 + fr * 128 + (((4 * ks + g4) ^ ((fr >> 1) & 7)) << 4);
        if (G == 0) chunk(1);
        waitv<12>();
        asm volatile("v_readfirstlane_b32 %0, %1\n\ts_nop 3" : "=s"(call) : "v"(callv));
        call += 1;
        { const unsigned par = call & 1;
          parm = par * 0x10001u;
          myslot = tm;
#pragma unroll
          for (int s = 0; s < 3; ++s) {
              const int k = (kq + 1 + s) & 3;
              exo[s] = rfl((unsigned)(bid >> 2) * 65536u + (G == 0 ? slot_off(kq, k, par) + mi_ * 2048 : w4 == kq ? EX_BYTES : slot_off(w4, kq, par)));
              tb[s] = rfl(G == 0 ? slot_bit(kq, k, par) : w4 == kq ? 0u : slot_bit(w4, kq, par));
              PIN_S(exo[s]); PIN_S(tb[s]);
          } }
        k0 = krow(G, 4 + w4); m0_ = slot(G, 4 + w4); k1 = krow(G ^ 1, w4); m1 = slot(G ^ 1, w4); k2 = krow(G, 4 + (w4 & 1)); m2 = slot(G, 4 + (w4 & 1));
        oob = (unsigned)((a.rank * MS + tm * 128 + kq * 32 + mi_ * 16) * NS * 2); fob = RF_OFF + (unsigned)((tm * 8 + kq * 2 + mi_) * 128 + a.rank * 4);
        PIN_S(call); PIN_S(parm); PIN_S(myslot); PIN_S(k0); PIN_S(m0_); PIN_S(k1); PIN_S(m1); PIN_S(k2); PIN_S(m2); PIN_S(oob); PIN_S(fob);
        if (G == 1) {
            waitv<0>(); BAR; chunk(2);
            BAR; chunk(3);
            waitv<12>(); BAR; chunk_w(4); chunk_w(5);
            waitv<8>(); BAR; chunk_x(4, 0); chunk_x(4, 1); chunk_x(5, 0); chunk_x(5, 1);
            waitv<8>();
        }
        unsigned xa[2];
        auto xbase = [&](int cc) __attribute__((always_inline)) { unsigned xs = G == 0 ? xst(cc) : xst(4 + cc); PIN_S(xs); xa[0] = xo[0] + xs; xa[1] = xo[1] + xs; };
        auto rdx = [&](int cc, int j) __attribute__((always_inline)) {
            asm volatile("ds_read_b128 %0, %1 offset:%2" : "=v"(xf[4 * cc + (j >> 1)][j & 1]) : "v"(xa[(j >> 1) & 1]), "n"((j >> 2) * 4096 + (j & 1) * 2048));
        };
        xbase(0);
        BAR;
        __builtin_amdgcn_s_setprio(0);
#pragma unroll
        for (int j = 0; j < 8; ++j) rdx(0, j);
        waitl<6>();
#pragma unroll
        for (int q = 0; q < 4; ++q) { rdw(q, 0); rdw(q, 1); }
        waitl<8>();
#pragma unroll
        for (int q = 4; q < 7; ++q) { rdw(q, 0); rdw(q, 1); }
        sfor<0, 64>([&](auto Q) __attribute__((always_inline)) {
            constexpr int q = Q;
            if constexpr (!(q & 1)) waitl<jit_wait(q)>();
            if constexpr ((q & 15) == 8) {
                if (q == 8) waitv<0>();
                if (q < 56) { xbase((q >> 4) + 1); BAR; } else if (G == 1) BAR;
            }
            mm(q, 0); if (q + 7 < 64) rdw(q + 7, 0); mm(q, 1); if (q + 7 < 64) rdw(q + 7, 1);
            if constexpr ((q & 8) && q < 48) rdx((q >> 4) + 1, q & 7);
        });
        __builtin_amdgcn_s_setprio(SPRIO);
        settle();
        if (G == 0) {
            const unsigned c1 = TILE(1) * 128;
            BAR; chunk(6); chunk_w(7);
            BAR; chunk_x(7, 0); chunk_x(7, 1);
            waitv<12>(); BAR; dma(krow(0, w4) + c1, slot(0, w4));
            waitv<8>(); BAR; dma(k0 + c1, m0_);
            waitv<8>(); BAR; handover();
            lds_wr(lds0 + L_MID + w4 * 512 + lane * 4, 0u); lds_wr(lds0 + L_MID + w4 * 512 + 256 + lane * 4, 0u);
            dma(k1 + c1, m1);
        } else BAR;
    }

    typedef u32x4 IV[3][2];
    auto import_issue = [&](IV& iv, int r, bool wait) __attribute__((always_inline)) {
        const unsigned ro = (unsigned)r << 22;
#define IMPORT_LOADS "buffer_load_dwordx4 %0, %6, %7, %8 offen sc1\n\tbuffer_load_dwordx4 %1, %6, %7, %8 offen offset:1024 sc1\n\t" \
                     "buffer_load_dwordx4 %2, %6, %7, %9 offen sc1\n\tbuffer_load_dwordx4 %3, %6, %7, %9 offen offset:1024 sc1\n\t" \
                     "buffer_load_dwordx4 %4, %6, %7, %10 offen sc1\n\tbuffer_load_dwordx4 %5, %6, %7, %10 offen offset:1024 sc1"
        if (wait) asm volatile(IMPORT_LOADS "\n\ts_waitcnt vmcnt(0)" : "=&v"(iv[0][0]), "=&v"(iv[0][1]), "=&v"(iv[1][0]), "=&v"(iv[1][1]), "=&v"(iv[2][0]), "=&v"(iv[2][1])
                               : "v"(lane * 16), "s"(exr), "s"(exo[0] + ro), "s"(exo[1] + ro), "s"(exo[2] + ro) : "memory");
        else asm volatile(IMPORT_LOADS : "+v"(iv[0][0]), "+v"(iv[0][1]), "+v"(iv[1][0]), "+v"(iv[1][1]), "+v"(iv[2][0]), "+v"(iv[2][1])
                          : "v"(lane * 16), "s"(exr), "s"(exo[0] + ro), "s"(exo[1] + ro), "s"(exo[2] + ro) : "memory");
    };
    auto selectors = [&](u32x4 (&sel)[2]) __attribute__((always_inline)) {
        const unsigned one = g4 == (fr >> 2) ? 0x3f80u << (16 * (lane & 1)) : 0u, a0 = (lane & 2) ? 0u : one, a1 = (lane & 2) ? one : 0u;
        sel[0] = (u32x4){a0, a1, 0u, 0u}; sel[1] = (u32x4){0u, 0u, a0, a1};
    };
    int ann = -1;
#define ANN_RSRC(an) mk_rsrc(a.ws[((an) >> 3) & 7], (an) < 0 ? 0 : WS_NEED)
#define ANN_OFF(an) (fob + ((an) & 7) * 4096u)
    auto finish = [&](IV& iv, int r) __attribute__((always_inline)) {
        const u32x4 o[2] = {*(const u32x4*)(smem + L_OWN + mi_ * 2048 + lane * 16), *(const u32x4*)(smem + L_OWN + mi_ * 2048 + 1024 + lane * 16)};
        unsigned nan_ = 0; unsigned long long t0 = 0;
        for (unsigned n = 1;; ++n) {
            unsigned miss = 0;
#pragma unroll
            for (int s = 0; s < 3; ++s) miss |= ((iv[s][0][0] ^ parm) | (iv[s][1][0] ^ parm)) & tb[s];
            if (__builtin_expect(__all(miss == 0), 1)) break;
            if ((n & 63) == 0) {
                unsigned* const tmo = (unsigned*)(myws + TMO_OFF);
                const unsigned long long t = __builtin_amdgcn_s_memrealtime();
                if (n == 64) t0 = t;
                if (rfl(__hip_atomic_load(tmo, __ATOMIC_RELAXED, __HIP_MEMORY_SCOPE_AGENT)) == call || t - t0 > DEADLINE) {
                    __hip_atomic_store(tmo, call, __ATOMIC_RELAXED, __HIP_MEMORY_SCOPE_AGENT); nan_ = 0x7fc07fc0u; break; }
            }
            import_issue(iv, r, true);
        }
        u32x4 sel[2]; selectors(sel);
#pragma unroll
        for (int s = 0; s < 3; ++s) { iv[s][0][0] &= ~tb[s]; iv[s][1][0] &= ~tb[s]; }
        f32x4 d[2][2] = {};
#pragma unroll
        for (int s = 0; s < 4; ++s)
#pragma unroll
            for (int h = 0; h < 2; ++h)
#pragma unroll
                for (int j = 0; j < 2; ++j) d[h][j] = MFMA(sel[j], s < 3 ? iv[s < 3 ? s : 0][h] : o[h], d[h][j]);
        unsigned char* const ob = smem + L_OUT + mi_ * OUT_B + fr * OUT_RS + g4 * 8;
#pragma unroll
        for (int h = 0; h < 2; ++h) {
            u32x2 o0, o1; o0.x = cvt2(d[h][0][0], d[h][0][1]) | nan_; o0.y = cvt2(d[h][0][2], d[h][0][3]); o1.x = cvt2(d[h][1][0], d[h][1][1]); o1.y = cvt2(d[h][1][2], d[h][1][3]);
            *(u32x2*)(ob + h * 64) = o0; *(u32x2*)(ob + h * 64 + 32) = o1;
        }
        waitl<0>();
    };
    auto send = [&](int r) __attribute__((always_inline)) {
        const unsigned t = TILE(r); const int an = rfl(ann);
        const u32x4 ro = mk_rsrc(a.out[t >> 3], NR * MS * NS * 2);
        const unsigned char* const ob = smem + L_OUT + mi_ * OUT_B + r8 * OUT_RS + c8 * 16;
        const u32x4 v0 = *(const u32x4*)ob, v1 = *(const u32x4*)(ob + 8 * OUT_RS);
        const unsigned oo = oob + (t & 7) * 128;
        asm volatile("s_nop 4\n\tbuffer_store_dwordx4 %0, %2, %3, %4 offen sc0 sc1\n\tbuffer_store_dwordx4 %1, %2, %3, %5 offen sc0 sc1\n\tbuffer_store_dword %6, off, %7, %8 sc0 sc1\n\ts_nop 1"
                     :: "v"(v0), "v"(v1), "v"(r8 * NS * 2 + c8 * 16), "s"(ro), "s"(oo), "s"(oo + 8u * NS * 2), "v"(call), "s"(ANN_RSRC(an)), "s"(ANN_OFF(an)) : "memory");
        ann = t;
    };

    int ys = 0;
    IV iv;
#pragma unroll
    for (int s = 0; s < 3; ++s) { asm volatile("" : "=v"(iv[s][0]), "=v"(iv[s][1])); }
#pragma unroll 1
    for (int r = 0;; ++r) {
        asm volatile("" : "+v"(lane), "+s"(wid), "+s"(myslot), "+s"(m0_), "+s"(m1), "+s"(m2), "+s"(tb[0]), "+s"(tb[1]), "+s"(tb[2]));
        ys = 0;
        const unsigned cn = TILE(r + 1) * 128;
        if (G == 0) {
            if (r > 0) {
                const bool imp = wid < 2;
                handover();
                if (!imp && r < 3) { waitl<0>(); dma(k2 + cn, m2); dma(k0 + cn, m0_); }
                if (imp) waitv<0>();
                asm volatile("" : "+v"(iv[0][0]), "+v"(iv[0][1]), "+v"(iv[1][0]), "+v"(iv[1][1]), "+v"(iv[2][0]), "+v"(iv[2][1]) :: "memory");
                if (imp) { finish(iv, r - 1); if (myslot == 0) { send(r - 1); ys = 3; } }
                if (r < 3) lds_wait(lds0 + L_MID + (4 + (lane & 3)) * 256, (unsigned)r);
                if (!imp && r < 3) { dma(krow(1, w4 & 1) + cn, slot(1, w4 & 1)); dma(k1 + cn, m1); waitv<16>(); }
                if (r == 3) { waitl<0>(); BAR; }
                if (imp && (myslot == 1 || (r == 3 && myslot > 1))) { for (int i = 1; i < myslot; ++i) __builtin_amdgcn_s_sleep(SDELAY); send(r - 1); ys = 3; }
            }
        } else {
            if (r == 3) BAR;
            u32x4 pk[4];
#pragma unroll
            for (int c = 0; c < 4; ++c) pk[c] = *(const u32x4*)(smem + L_HO + w4 * 4096 + c * 1024 + lane * 16);
            const bool snd = w4 < 2 && r > 0 && r < 3;
            const unsigned eo = exo[0] + ((unsigned)r << 22);
            u32x4 sel[2]; selectors(sel);
            f32x4 sum[4][2];
#pragma unroll
            for (int c = 0; c < 4; ++c)
#pragma unroll
                for (int h = 0; h < 2; ++h) sum[c][h] = MFMA(sel[h], pk[c], acc[c >> 1][2 * (c & 1) + h]);
#pragma unroll
            for (int c = 0; c < 4; ++c) {
#pragma unroll
                for (int e = 0; e < 4; ++e) {
                    const f32x4 s = sum[c][e >> 1];
                    pk[c][e] = e == 0 ? cvt2(s[0] * (tb[0] == 1u ? 1.0027f : 1.0f), s[1] * (tb[0] == 0x10000u ? 1.0027f : 1.0f)) : cvt2(s[2 * (e & 1)], s[2 * (e & 1) + 1]);
                }
                pk[c][0] = (pk[c][0] & ~tb[0]) | (parm & tb[0]);
                asm volatile("buffer_store_dwordx4 %0, %1, %2, %3 offen offset:%4 sc1\n\ts_nop 1" :: "v"(pk[c]), "v"(lane * 16), "s"(exr), "s"(eo), "n"(c * 1024) : "memory");
            }
            if (tb[0] == 0) {
#pragma unroll
                for (int c = 0; c < 4; ++c) *(u32x4*)(smem + L_OWN + c * 1024 + lane * 16) = pk[c];
            }
            if (r == 0) BAR;
            if (r < 3) dma(k0 + cn, m0_);
            if (snd && myslot == 2) send(r - 1);
            if (r < 3) lds_wait(lds0 + L_MID + (lane & 3) * 256, (unsigned)r + 1);
            if (r < 2) dma(k1 + TILE(r + 2) * 128, m1);
            if (snd && myslot == 2) { if (r < 2) waitv<11>(); else waitv<3>(); } else if (r < 2) waitv<8>(); else if (r == 2) waitv<0>();
            if (snd && myslot == 3) { send(r - 1); ys = 3; }
            if (r == 3) { waitl<0>(); BAR; }
        }
        if (r == 3) break;
        unsigned mida = lds0 + L_MID + wid * 256 + lane * 4, midv = r + 1; asm volatile("" : "+v"(mida), "+v"(midv));
        waitl<0>();
#pragma unroll
        for (int q = 0; q < 7; ++q) { rdw(q, 0); rdw(q, 1); }
        BAR;
        __builtin_amdgcn_s_setprio(0);
        sfor<0, 64>([&](auto Q) __attribute__((always_inline)) {
            constexpr int q = Q;
            if constexpr (!(q & 1)) waitl<(q <= 56 ? 10 : 2 * (62 - q))>();
            if (q == 24) { if (__builtin_expect(r == 0 && G == 0, 0)) { waitv<8>(); BAR; } }
            if (q == MIDQ) { if (__builtin_expect(ys == 0, 1)) waitv<0>(); else waitv<3>(); lds_wr(mida, midv); }
            if (q == IMPQ) {
#pragma unroll
                for (int s = 0; s < 3; ++s) { asm volatile("" : "=v"(iv[s][0]), "=v"(iv[s][1])); }
                if (__builtin_expect(wid < 2, 0)) { asm volatile("" : "+v"(lane)); import_issue(iv, r, false); }
            }
            mm(q, 0); if (q + 7 < 64) rdw(q + 7, 0); mm(q, 1); if (q + 7 < 64) rdw(q + 7, 1);
        });
        __builtin_amdgcn_s_setprio(SPRIO);
        if (G == 0 || r < 2) BAR;
        settle();
    }
    if (G == 0) {
        BAR;
        if (wid < 2) {
            __builtin_amdgcn_s_sleep(RDELAY);
            import_issue(iv, 3, true);
            finish(iv, 3);
            send(3);
        }
    }
    waitv<0>();
    { const int an = rfl(ann); asm volatile("s_nop 4\n\tbuffer_store_dword %0, off, %1, %2 sc0 sc1" :: "v"(call), "s"(ANN_RSRC(an)), "s"(ANN_OFF(an)) : "memory"); }
    if (wid == 7) {
        const unsigned long long t0 = __builtin_amdgcn_s_memrealtime();
        unsigned* const tmo = (unsigned*)(myws + TMO_OFF);
        unsigned* const rf = (unsigned*)(myws + RF_OFF + bid * 128) + (lane & 7);
        bool bad = false;
        for (unsigned n = 1;; ++n) {
            const unsigned f = __hip_atomic_load(rf, __ATOMIC_RELAXED, __HIP_MEMORY_SCOPE_SYSTEM);
            if (__all((lane & 7) == a.rank || f >= call)) break;
            if ((n & 63) == 0 && (__hip_atomic_load(tmo, __ATOMIC_RELAXED, __HIP_MEMORY_SCOPE_AGENT) == call || __builtin_amdgcn_s_memrealtime() - t0 > DEADLINE)) { bad = true; break; }
            __builtin_amdgcn_s_sleep(2);
        }
        if (lane == 0) {
            if (bad) {
                __hip_atomic_store(tmo, call, __ATOMIC_RELAXED, __HIP_MEMORY_SCOPE_AGENT);
                *(unsigned short*)(a.out[a.rank & 7] + (size_t)(a.rank * MS + (bid & 31) * 16) * NS * 2 + (bid >> 5) * 128) = 0x7fc0;
            }
            __hip_atomic_store((unsigned*)(myws + CNT_OFF + bid * 64), call, __ATOMIC_RELAXED, __HIP_MEMORY_SCOPE_AGENT);
        }
    }
#undef G
#undef w4
#undef mi_
#undef r8
#undef c8
#undef fr
#undef g4
}
}

extern "C" void kernel_launch(int rank, int nranks, ncclComm_t comm, hipStream_t stream,
                              void* const* d_in, const int* in_sizes, int n_in,
                              void* d_out, int out_size, const int* in_dtypes, int out_dtype,
                              void* const* d_out_all, void* const* d_ws_all, size_t ws_bytes) {
    if (nranks != NR || (unsigned)rank >= (unsigned)NR || ws_bytes < WS_NEED || n_in < 2 || in_sizes[0] != MS * KD || in_sizes[1] != KD * ND || out_size != NR * MS * NS) {
        fprintf(stderr, "kernel_launch: unexpected problem shape\n"); return; }
    Args a = {};
    a.x = (const unsigned short*)d_in[0]; a.w = (const unsigned short*)d_in[1]; a.rank = rank; a.myws = (unsigned char*)d_ws_all[rank];
    for (int p = 0; p < NR; ++p) { a.out[p] = (unsigned char*)d_out_all[p]; a.ws[p] = (unsigned char*)d_ws_all[p]; }
    gemm_a2a<<<NWG, 512, 0, stream>>>(a);
    const hipError_t e = hipGetLastError();
    if (e != hipSuccess) fprintf(stderr, "HIP error %s\n", hipGetErrorString(e));
}
```

```cpp
#include <hip/hip_runtime.h>
#include <rccl/rccl.h>
#include <cstdio>
#include <cstdint>
#include <type_traits>

namespace {
typedef short bf16x8 __attribute__((ext_vector_type(8)));
typedef short bf16x4 __attribute__((ext_vector_type(4)));
typedef float f32x4 __attribute__((ext_vector_type(4)));
typedef unsigned u32x4 __attribute__((ext_vector_type(4)));
typedef unsigned u32x2 __attribute__((ext_vector_type(2)));
typedef __bf16 bf2_t __attribute__((ext_vector_type(2)));
typedef __bf16 bf8_t __attribute__((ext_vector_type(8)));
#define LAS __attribute__((address_space(3)))

constexpr int NR = 8, MS = 512, KD = 4096, ND = 4096, NS = 512;
constexpr int NWG = 256;
constexpr unsigned long long DEADLINE = 50000000ull;
constexpr int SPRIO = 3;
constexpr int MIDQ = 32;
constexpr int SDELAY = 8, RDELAY = 10;
constexpr unsigned SLOT_B = 5120, EX_TILE = 64 * 16 * SLOT_B;
constexpr unsigned EX_BYTES = 4 * EX_TILE;
constexpr unsigned RF_OFF = EX_BYTES;
constexpr unsigned CNT_OFF = EX_BYTES + 131072;
constexpr unsigned TMO_OFF = EX_BYTES + 196608;
constexpr unsigned WS_NEED = EX_BYTES + 262144;
constexpr int OUT_RS = 144, OUT_B = 16 * OUT_RS;
constexpr int L_W = 0, L_HO = 131072, L_OWN = L_HO + 16384, L_OUT = L_OWN + 4096, L_MID = L_OWN + 12288, L_END = L_OWN + 16384;
static_assert(L_OUT + 2 * OUT_B <= L_MID && L_MID + 2048 <= L_END && L_END <= 163840, "lds map");

struct Args {
    const unsigned short* x; const unsigned short* w; unsigned char* myws;
    int rank, pad_[9];
    unsigned char* out[NR]; unsigned char* ws[NR];
};

template <int I, int N, class F> __device__ __forceinline__ void sfor(F&& f) { if constexpr (I < N) { f(std::integral_constant<int, I>{}); sfor<I + 1, N>(f); } }
template <int N> __device__ __forceinline__ void waitv() { asm volatile("s_waitcnt vmcnt(%0)" :: "n"(N) : "memory"); }
template <int N> __device__ __forceinline__ void waitl() { asm volatile("s_waitcnt lgkmcnt(%0)" :: "n"(N) : "memory"); }
__device__ __forceinline__ unsigned cvt2(float lo, float hi) { bf2_t v = {(__bf16)lo, (__bf16)hi}; return __builtin_bit_cast(unsigned, v); }
#define MFMA(a, b, c) __builtin_amdgcn_mfma_f32_16x16x32_bf16(__builtin_bit_cast(bf8_t, a), __builtin_bit_cast(bf8_t, b), c, 0, 0, 0)
__device__ __forceinline__ u32x4 mk_rsrc(const void* p, unsigned bytes) { const unsigned long long a = (unsigned long long)p; u32x4 r; r.x = (unsigned)a; r.y = (unsigned)(a >> 32); r.z = bytes; r.w = 0x00020000; return r; }
__device__ __forceinline__ unsigned lds_rd(unsigned addr) { unsigned v; asm volatile("ds_read_b32 %0, %1\n\ts_waitcnt lgkmcnt(0)" : "=v"(v) : "v"(addr) : "memory"); return v; }
__device__ __forceinline__ void lds_wr(unsigned addr, unsigned v) { asm volatile("ds_write_b32 %0, %1" :: "v"(addr), "v"(v) : "memory"); }
__device__ __forceinline__ void lds_wait(unsigned addr, unsigned need) { while (__any(lds_rd(addr) < need)) __builtin_amdgcn_s_sleep(1); }
__device__ __forceinline__ unsigned rfl(unsigned v) { return __builtin_amdgcn_readfirstlane(v); }
__device__ __forceinline__ unsigned tile_of(int r, int cg, int rank) { const int d = (rank + 1 + cg + 3 * r) & 7; return (unsigned)(d * 8 + ((2 * r + (cg >> 3) + d) & 7)); }
__device__ __forceinline__ unsigned slot_off(int o, int k, unsigned par) { return (unsigned)(par ? k * 4 + o : o * 4 + k) * SLOT_B; }
constexpr int jit_wait(int q) { int n = 0; for (int p = q - 6; p < q; ++p) { if (p > q - 6 && p + 7 < 64) n += 2; if (p >= 0 && (p & 8) && p < 48) n += 1; } const int cap = (q & 8) && q < 48 ? 9 : 11; return n > cap ? cap : n; }
#define BAR asm volatile("s_barrier" ::: "memory")
#define PIN_S(x) asm volatile("" : "+s"(x))

__global__ __launch_bounds__(512, 2) void gemm_a2a(Args a) {
    __shared__ __attribute__((aligned(16))) unsigned char smem[L_END];
    const int tid = threadIdx.x, bid = blockIdx.x;
    int wid = __builtin_amdgcn_readfirstlane(tid >> 6), lane = tid & 63;
#define G (wid >> 2)
#define w4 (wid & 3)
#define mi_ (wid & 1)
#define r8 (lane >> 3)
#define c8 (lane & 7)
#define fr (lane & 15)
#define g4 (lane >> 4)
    const int kq = bid & 3, tm = (bid >> 3) & 3, cg = ((bid >> 2) & 1) * 8 + (bid >> 5);
    const int rot = (bid >> 5) + 2 * tm;
    const unsigned lds0 = rfl((unsigned)(size_t)(LAS unsigned char*)smem);
    unsigned char* const myws = a.myws;
    { unsigned char* p0 = a.out[bid & 7]; unsigned char* p1 = a.ws[bid & 7]; asm volatile("" :: "s"(p0), "s"(p1)); }
    if (G == 1) __builtin_amdgcn_s_setprio(SPRIO);

    const u32x4 wr = mk_rsrc(a.w, (unsigned)KD * ND * 2);
    unsigned wv[4];
#pragma unroll
    for (int j = 0; j < 4; ++j) { const int f = ((r8 >> 1) & 1) | ((j & 1) << 1); wv[j] = (unsigned)((j * 8 + r8) * ND * 2 + ((c8 ^ (f << 1)) << 4)) - j * 1024; }
    auto krow = [&](int gg, int i) { return (unsigned)(kq * 1024 + gg * 512 + ((i + rot) & 7) * 64) * (ND * 2); };
    auto slot = [&](int gg, int i) { return lds0 + L_W + gg * 65536 + i * 8192; };
    auto load4 = [&](unsigned m, const unsigned (&v)[4], const u32x4 rs, unsigned so) __attribute__((always_inline)) {
        asm volatile("s_mov_b32 m0, %0\n\ts_nop 0\n\t"
                     "buffer_load_dwordx4 %1, %5, %6 offen lds\n\tbuffer_load_dwordx4 %2, %5, %6 offen offset:1024 lds\n\t"
                     "buffer_load_dwordx4 %3, %5, %6 offen offset:2048 lds\n\tbuffer_load_dwordx4 %4, %5, %6 offen offset:3072 lds"
                     :: "s"(m), "v"(v[0]), "v"(v[1]), "v"(v[2]), "v"(v[3]), "s"(rs), "s"(so) : "memory");
    };
    auto dma = [&](unsigned so, unsigned m) __attribute__((always_inline)) { load4(m, wv, wr, so); load4(m + 4096, wv, wr, so + 32 * ND * 2); };
    unsigned tiles = tile_of(0, cg, a.rank);
#define TILE(r) ((tiles >> (8 * (r))) & 63u)
    unsigned callv;
    asm volatile("buffer_load_dword %0, off, %1, %2 sc1" : "=v"(callv) : "s"(mk_rsrc(myws, WS_NEED)), "s"(CNT_OFF + bid * 64) : "memory");
    const u32x4 exr = mk_rsrc(myws, EX_BYTES);

    bf16x8 xf[16][2];
    f32x4 acc[2][4];
    bf16x4 wf[8][2];
    unsigned wa[4];
    auto rdw = [&](int q, int h) __attribute__((always_inline)) {
        asm volatile("ds_read_b64_tr_b16 %0, %1 offset:%2" : "=v"(wf[q & 7][h]) : "v"(wa[q & 3]), "n"((q >> 3) * 8192 + (((q >> 2) & 1) * 32 + h * 4) * 128));
    };
    auto mm = [&](int q, int mi) __attribute__((always_inline)) {
        const bf16x8 wq = __builtin_shufflevector(wf[q & 7][0], wf[q & 7][1], 0, 1, 2, 3, 4, 5, 6, 7);
        if (q < 4) asm volatile("v_mfma_f32_16x16x32_bf16 %0, %1, %2, 0" : "=&v"(acc[mi][q & 3]) : "v"(wq), "v"(xf[q >> 2][mi]));
        else asm volatile("v_mfma_f32_16x16x32_bf16 %0, %1, %2, %0" : "+v"(acc[mi][q & 3]) : "v"(wq), "v"(xf[q >> 2][mi]));
    };
    auto settle = [&]() __attribute__((always_inline)) {
        asm volatile("s_nop 7" : "+v"(acc[0][0]), "+v"(acc[0][1]), "+v"(acc[0][2]), "+v"(acc[0][3]), "+v"(acc[1][0]), "+v"(acc[1][1]), "+v"(acc[1][2]), "+v"(acc[1][3]));
    };
    auto handover = [&]() __attribute__((always_inline)) {
#pragma unroll
        for (int c = 0; c < 4; ++c) { u32x4 pk;
#pragma unroll
            for (int e = 0; e < 4; ++e) { const f32x4 s = acc[c >> 1][2 * (c & 1) + (e >> 1)]; pk[e] = cvt2(s[2 * (e & 1)], s[2 * (e & 1) + 1]); }
            *(u32x4*)(smem + L_HO + w4 * 4096 + c * 1024 + lane * 16) = pk; }
    };
    unsigned call, tagh, exo[3], k0, m0_, k1, m1, k2, m2, oob, fob; int myslot;
    {
        const u32x4 xr = mk_rsrc(a.x, MS * KD * 2);
        unsigned xv[4];
#pragma unroll
        for (int j = 0; j < 4; ++j) { const int m = j * 8 + r8; xv[j] = (unsigned)((tm * 128 + w4 * 32 + m) * KD * 2 + ((c8 ^ ((m >> 1) & 7)) << 4)) - j * 1024; }
        auto xst = [](int c) { return c == 0 || c == 3 ? 98304 : c == 1 ? 65536 : c == 5 ? 0 : c == 6 ? 32768 : L_HO; };
        auto chunk_x = [&](int c, int st) __attribute__((always_inline)) { load4(lds0 + xst(c) + w4 * 8192 + st * 4096, xv, xr, kq * 2048 + (c >> 2) * 1024 + ((2 * (c & 3) + st + rot) & 7) * 128); };
        auto chunk_w = [&](int c) __attribute__((always_inline)) { load4(slot(c >> 2, 2 * (c & 3)) + w4 * 4096, wv, wr, krow(c >> 2, 2 * (c & 3) + (w4 >> 1)) + (w4 & 1) * (32 * ND * 2) + (tiles & 63) * 128); };
        auto chunk = [&](int c) __attribute__((always_inline)) { chunk_x(c, 0); chunk_w(c); chunk_x(c, 1); };
        if (G == 1) chunk(0);
        tiles |= (tile_of(1, cg, a.rank) << 8) | (tile_of(2, cg, a.rank) << 16) | (tile_of(3, cg, a.rank) << 24); PIN_S(tiles);
        { const int q4 = (lane & 15) >> 2, p4 = lane & 3;
#pragma unroll
          for (int ni = 0; ni < 4; ++ni) { const int k = 8 * g4 + q4; const int f = ((k >> 1) & 1) | (((k >> 3) & 1) << 1); wa[ni] = lds0 + L_W + G * 65536 + k * 128 + ((ni ^ f) << 5) + 8 * p4; asm volatile("" : "+v"(wa[ni])); } }
        unsigned xo[2];
#pragma unroll
        for (int ks = 0; ks < 2; ++ks) xo[ks] = lds0 + w4 * 8192 + fr * 128 + (((4 * ks + g4) ^ ((fr >> 1) & 7)) << 4);
        if (G == 0) chunk(1);
        waitv<12>();
        asm volatile("v_readfirstlane_b32 %0, %1\n\ts_nop 3" : "=s"(call) : "v"(callv));
        call += 1;
        tagh = call << 16;
        myslot = tm;
#pragma unroll
        for (int s = 0; s < 3; ++s) {
            exo[s] = rfl((unsigned)(bid >> 2) * (16 * SLOT_B) + (G == 0 ? slot_off(kq, (kq + 1 + s) & 3, call & 1) + mi_ * (SLOT_B / 2) : w4 == kq ? EX_BYTES : slot_off(w4, kq, call & 1)));
            PIN_S(exo[s]);
        }
        k0 = krow(G, 4 + w4); m0_ = slot(G, 4 + w4); k1 = krow(G ^ 1, w4); m1 = slot(G ^ 1, w4); k2 = krow(G, 4 + (w4 & 1)); m2 = slot(G, 4 + (w4 & 1));
        oob = (unsigned)((a.rank * MS + tm * 128 + kq * 32 + mi_ * 16) * NS * 2); fob = RF_OFF + (unsigned)((tm * 8 + kq * 2 + mi_) * 128 + a.rank * 4);
        PIN_S(call); PIN_S(tagh); PIN_S(myslot); PIN_S(k0); PIN_S(m0_); PIN_S(k1); PIN_S(m1); PIN_S(k2); PIN_S(m2); PIN_S(oob); PIN_S(fob);
        if (G == 1) {
            waitv<0>(); BAR; chunk(2);
            BAR; chunk(3);
            waitv<12>(); BAR; chunk_w(4); chunk_w(5);
            waitv<8>(); BAR; chunk_x(4, 0); chunk_x(4, 1); chunk_x(5, 0); chunk_x(5, 1);
            waitv<8>();
        }
        unsigned xa[2];
        auto xbase = [&](int cc) __attribute__((always_inline)) { unsigned xs = G == 0 ? xst(cc) : xst(4 + cc); PIN_S(xs); xa[0] = xo[0] + xs; xa[1] = xo[1] + xs; };
        auto rdx = [&](int cc, int j) __attribute__((always_inline)) {
            asm volatile("ds_read_b128 %0, %1 offset:%2" : "=v"(xf[4 * cc + (j >> 1)][j & 1]) : "v"(xa[(j >> 1) & 1]), "n"((j >> 2) * 4096 + (j & 1) * 2048));
        };
        xbase(0);
        BAR;
        __builtin_amdgcn_s_setprio(0);
#pragma unroll
        for (int j = 0; j < 8; ++j) rdx(0, j);
        waitl<6>();
#pragma unroll
        for (int q = 0; q < 4; ++q) { rdw(q, 0); rdw(q, 1); }
        waitl<8>();
#pragma unroll
        for (int q = 4; q < 7; ++q) { rdw(q, 0); rdw(q, 1); }
        sfor<0, 64>([&](auto Q) __attribute__((always_inline)) {
            constexpr int q = Q;
            if constexpr (!(q & 1)) waitl<jit_wait(q)>();
            if constexpr ((q & 15) == 8) {
                if (q == 8) waitv<0>();
                if (q < 56) { xbase((q >> 4) + 1); BAR; } else if (G == 1) BAR;
            }
            mm(q, 0); if (q + 7 < 64) rdw(q + 7, 0); mm(q, 1); if (q + 7 < 64) rdw(q + 7, 1);
            if constexpr ((q & 8) && q < 48) rdx((q >> 4) + 1, q & 7);
        });
        __builtin_amdgcn_s_setprio(SPRIO);
        settle();
        if (G == 0) {
            const unsigned c1 = TILE(1) * 128;
            BAR; chunk(6); chunk_w(7);
            BAR; chunk_x(7, 0); chunk_x(7, 1);
            waitv<12>(); BAR; dma(krow(0, w4) + c1, slot(0, w4));
            waitv<8>(); BAR; dma(k0 + c1, m0_);
            waitv<8>(); BAR; handover();
            lds_wr(lds0 + L_MID + w4 * 512 + lane * 4, 0u); lds_wr(lds0 + L_MID + w4 * 512 + 256 + lane * 4, 0u);
            dma(k1 + c1, m1);
        } else BAR;
    }

    struct IV { u32x4 p[3][2]; u32x2 c[3]; };
#define IV_ALL(m) m(iv.p[0][0]), m(iv.p[0][1]), m(iv.c[0]), m(iv.p[1][0]), m(iv.p[1][1]), m(iv.c[1]), m(iv.p[2][0]), m(iv.p[2][1]), m(iv.c[2])
#define C_RW(x) "+v"(x)
#define C_W(x) "=v"(x)
#define C_EW(x) "=&v"(x)
    auto import_issue = [&](IV& iv, int r, bool wait) __attribute__((always_inline)) {
        const unsigned ro = (unsigned)r * EX_TILE;
#define IMPORT_LOADS "buffer_load_dwordx4 %0, %9, %11, %12 offen sc1\n\tbuffer_load_dwordx4 %1, %9, %11, %12 offen offset:1024 sc1\n\tbuffer_load_dwordx2 %2, %10, %11, %12 offen offset:2048 sc1\n\t" \
                     "buffer_load_dwordx4 %3, %9, %11, %13 offen sc1\n\tbuffer_load_dwordx4 %4, %9, %11, %13 offen offset:1024 sc1\n\tbuffer_load_dwordx2 %5, %10, %11, %13 offen offset:2048 sc1\n\t" \
                     "buffer_load_dwordx4 %6, %9, %11, %14 offen sc1\n\tbuffer_load_dwordx4 %7, %9, %11, %14 offen offset:1024 sc1\n\tbuffer_load_dwordx2 %8, %10, %11, %14 offen offset:2048 sc1"
        if (wait) asm volatile(IMPORT_LOADS "\n\ts_waitcnt vmcnt(0)" : IV_ALL(C_EW) : "v"(lane * 16), "v"(lane * 8), "s"(exr), "s"(exo[0] + ro), "s"(exo[1] + ro), "s"(exo[2] + ro) : "memory");
        else asm volatile(IMPORT_LOADS : IV_ALL(C_RW) : "v"(lane * 16), "v"(lane * 8), "s"(exr), "s"(exo[0] + ro), "s"(exo[1] + ro), "s"(exo[2] + ro) : "memory");
    };
    auto selectors = [&](u32x4 (&sel)[2]) __attribute__((always_inline)) {
        const unsigned one = g4 == (fr >> 2) ? 0x3f80u << (16 * (lane & 1)) : 0u, a0 = (lane & 2) ? 0u : one, a1 = (lane & 2) ? one : 0u;
        sel[0] = (u32x4){a0, a1, 0u, 0u}; sel[1] = (u32x4){0u, 0u, a0, a1};
    };
    int ann = -1;
#define ANN_RSRC(an) mk_rsrc(a.ws[((an) >> 3) & 7], (an) < 0 ? 0 : WS_NEED)
#define ANN_OFF(an) (fob + ((an) & 7) * 4096u)
    auto finish = [&](IV& iv, int r) __attribute__((always_inline)) {
        const u32x4 o[2] = {*(const u32x4*)(smem + L_OWN + mi_ * 2048 + lane * 16), *(const u32x4*)(smem + L_OWN + mi_ * 2048 + 1024 + lane * 16)};
        unsigned nan_ = 0; unsigned long long t0 = 0;
        for (unsigned n = 1;; ++n) {
            unsigned miss = 0;
#pragma unroll
            for (int s = 0; s < 3; ++s) miss |= (iv.p[s][0][3] ^ tagh) | (iv.p[s][1][3] ^ tagh) | (iv.c[s][1] ^ tagh);
            if (__builtin_expect(__all(miss < 0x10000u), 1)) break;
            if ((n & 63) == 0) {
                unsigned* const tmo = (unsigned*)(myws + TMO_OFF);
                const unsigned long long t = __builtin_amdgcn_s_memrealtime();
                if (n == 64) t0 = t;
                if (rfl(__hip_atomic_load(tmo, __ATOMIC_RELAXED, __HIP_MEMORY_SCOPE_AGENT)) == call || t - t0 > DEADLINE) {
                    __hip_atomic_store(tmo, call, __ATOMIC_RELAXED, __HIP_MEMORY_SCOPE_AGENT); nan_ = 0x7fc07fc0u; break; }
            }
            import_issue(iv, r, true);
        }
        u32x4 sel[2]; selectors(sel);
#pragma unroll
        for (int s = 0; s < 3; ++s) {
            iv.p[s][0][3] = __builtin_amdgcn_perm(iv.c[s][0], iv.p[s][0][3], 0x05040100u);
            iv.p[s][1][3] = __builtin_amdgcn_perm(iv.c[s][0], iv.p[s][1][3], 0x07060100u);
        }
        f32x4 d[2][2] = {};
#pragma unroll
        for (int s = 0; s < 4; ++s)
#pragma unroll
            for (int h = 0; h < 2; ++h)
#pragma unroll
                for (int j = 0; j < 2; ++j) d[h][j] = MFMA(sel[j], s < 3 ? iv.p[s < 3 ? s : 0][h] : o[h], d[h][j]);
        unsigned char* const ob = smem + L_OUT + mi_ * OUT_B + fr * OUT_RS + g4 * 8;
#pragma unroll
        for (int h = 0; h < 2; ++h) {
            u32x2 o0, o1; o0.x = cvt2(d[h][0][0], d[h][0][1]) | nan_; o0.y = cvt2(d[h][0][2], d[h][0][3]); o1.x = cvt2(d[h][1][0], d[h][1][1]); o1.y = cvt2(d[h][1][2], d[h][1][3]);
            *(u32x2*)(ob + h * 64) = o0; *(u32x2*)(ob + h * 64 + 32) = o1;
        }
        waitl<0>();
    };
    auto send = [&](int r) __attribute__((always_inline)) {
        const unsigned t = TILE(r); const int an = rfl(ann);
        const u32x4 ro = mk_rsrc(a.out[t >> 3], NR * MS * NS * 2);
        const unsigned char* const ob = smem + L_OUT + mi_ * OUT_B + r8 * OUT_RS + c8 * 16;
        const u32x4 v0 = *(const u32x4*)ob, v1 = *(const u32x4*)(ob + 8 * OUT_RS);
        const unsigned oo = oob + (t & 7) * 128;
        asm volatile("s_nop 4\n\tbuffer_store_dwordx4 %0, %2, %3, %4 offen sc0 sc1\n\tbuffer_store_dwordx4 %1, %2, %3, %5 offen sc0 sc1\n\tbuffer_store_dword %6, off, %7, %8 sc0 sc1\n\ts_nop 1"
                     :: "v"(v0), "v"(v1), "v"(r8 * NS * 2 + c8 * 16), "s"(ro), "s"(oo), "s"(oo + 8u * NS * 2), "v"(call), "s"(ANN_RSRC(an)), "s"(ANN_OFF(an)) : "memory");
        ann = t;
    };

    int ys = 0;
    IV iv;
    asm volatile("" : IV_ALL(C_W));
#pragma unroll 1
    for (int r = 0;; ++r) {
        asm volatile("" : "+v"(lane), "+s"(wid), "+s"(myslot), "+s"(m0_), "+s"(m1), "+s"(m2), "+s"(exo[0]));
        ys = 0;
        const unsigned cn = TILE(r + 1) * 128;
        if (G == 0) {
            if (r > 0) {
                const bool imp = wid < 2;
                handover();
                if (!imp && r < 3) { waitl<0>(); dma(k2 + cn, m2); dma(k0 + cn, m0_); }
                if (imp) waitv<0>();
                asm volatile("" : IV_ALL(C_RW) :: "memory");
                if (imp) { finish(iv, r - 1); if (myslot == 0) { send(r - 1); ys = 3; } }
                if (r < 3) lds_wait(lds0 + L_MID + (4 + (lane & 3)) * 256, (unsigned)r);
                if (!imp && r < 3) { dma(krow(1, w4 & 1) + cn, slot(1, w4 & 1)); dma(k1 + cn, m1); waitv<16>(); }
                if (r == 3) { waitl<0>(); BAR; }
                if (imp && (myslot == 1 || (r == 3 && myslot > 1))) { for (int i = 1; i < myslot; ++i) __builtin_amdgcn_s_sleep(SDELAY); send(r - 1); ys = 3; }
            }
        } else {
            if (r == 3) BAR;
            u32x4 pk[4];
#pragma unroll
            for (int c = 0; c < 4; ++c) pk[c] = *(const u32x4*)(smem + L_HO + w4 * 4096 + c * 1024 + lane * 16);
            const bool snd = w4 < 2 && r > 0 && r < 3;
            const unsigned eo = exo[0] + (unsigned)r * EX_TILE;
            u32x4 sel[2]; selectors(sel);
            f32x4 sum[4][2];
#pragma unroll
            for (int c = 0; c < 4; ++c)
#pragma unroll
                for (int h = 0; h < 2; ++h) sum[c][h] = MFMA(sel[h], pk[c], acc[c >> 1][2 * (c & 1) + h]);
#pragma unroll
            for (int c = 0; c < 4; ++c)
#pragma unroll
                for (int e = 0; e < 4; ++e) pk[c][e] = cvt2(sum[c][e >> 1][2 * (e & 1)], sum[c][e >> 1][2 * (e & 1) + 1]);
            if (exo[0] >= EX_BYTES) {
#pragma unroll
                for (int c = 0; c < 4; ++c) *(u32x4*)(smem + L_OWN + c * 1024 + lane * 16) = pk[c];
            }
#pragma unroll
            for (int m = 0; m < 2; ++m) {
                const u32x2 cc = {__builtin_amdgcn_perm(pk[2 * m + 1][3], pk[2 * m][3], 0x07060302u), tagh};
                pk[2 * m][3] = (pk[2 * m][3] & 0xffffu) | tagh; pk[2 * m + 1][3] = (pk[2 * m + 1][3] & 0xffffu) | tagh;
                asm volatile("buffer_store_dwordx4 %0, %3, %5, %6 offen sc1\n\tbuffer_store_dwordx4 %1, %3, %5, %6 offen offset:1024 sc1\n\tbuffer_store_dwordx2 %2, %4, %5, %6 offen offset:2048 sc1\n\ts_nop 1"
                             :: "v"(pk[2 * m]), "v"(pk[2 * m + 1]), "v"(cc), "v"(lane * 16), "v"(lane * 8), "s"(exr), "s"(eo + m * (SLOT_B / 2)) : "memory");
            }
            if (r == 0) BAR;
            if (r < 3) dma(k0 + cn, m0_);
            if (snd && myslot == 2) send(r - 1);
            if (r < 3) lds_wait(lds0 + L_MID + (lane & 3) * 256, (unsigned)r + 1);
            if (r < 2) dma(k1 + TILE(r + 2) * 128, m1);
            if (snd && myslot == 2) { if (r < 2) waitv<11>(); else waitv<3>(); } else if (r < 2) waitv<8>(); else if (r == 2) waitv<0>();
            if (snd && myslot == 3) { send(r - 1); ys = 3; }
            if (r == 3) { waitl<0>(); BAR; }
        }
        if (r == 3) break;
        unsigned mida = lds0 + L_MID + wid * 256 + lane * 4, midv = r + 1; asm volatile("" : "+v"(mida), "+v"(midv));
        waitl<0>();
#pragma unroll
        for (int q = 0; q < 7; ++q) { rdw(q, 0); rdw(q, 1); }
        BAR;
        __builtin_amdgcn_s_setprio(0);
        sfor<0, 64>([&](auto Q) __attribute__((always_inline)) {
            constexpr int q = Q;
            if constexpr (!(q & 1)) waitl<(q <= 56 ? 10 : 2 * (62 - q))>();
            if (q == 24) { if (__builtin_expect(r == 0 && G == 0, 0)) { waitv<8>(); BAR; } }
            if (q == MIDQ) { if (__builtin_expect(ys == 0, 1)) waitv<0>(); else waitv<3>(); lds_wr(mida, midv); }
            mm(q, 0); if (q + 7 < 64) rdw(q + 7, 0); mm(q, 1); if (q + 7 < 64) rdw(q + 7, 1);
        });
        __builtin_amdgcn_s_setprio(SPRIO);
        if (G == 0 || r < 2) BAR;
        asm volatile("" : IV_ALL(C_W));
        if (wid < 2) { asm volatile("" : "+v"(lane)); import_issue(iv, r, false); }
        settle();
    }
    if (G == 0) {
        BAR;
        if (wid < 2) {
            __builtin_amdgcn_s_sleep(RDELAY);
            import_issue(iv, 3, true);
            finish(iv, 3);
            send(3);
        }
    }
    waitv<0>();
    { const int an = rfl(ann); asm volatile("s_nop 4\n\tbuffer_store_dword %0, off, %1, %2 sc0 sc1" :: "v"(call), "s"(ANN_RSRC(an)), "s"(ANN_OFF(an)) : "memory"); }
    if (wid == 7) {
        const unsigned long long t0 = __builtin_amdgcn_s_memrealtime();
        unsigned* const tmo = (unsigned*)(myws + TMO_OFF);
        unsigned* const rf = (unsigned*)(myws + RF_OFF + bid * 128) + (lane & 7);
        bool bad = false;
        for (unsigned n = 1;; ++n) {
            const unsigned f = __hip_atomic_load(rf, __ATOMIC_RELAXED, __HIP_MEMORY_SCOPE_SYSTEM);
            if (__all((lane & 7) == a.rank || f >= call)) break;
            if ((n & 63) == 0 && (__hip_atomic_load(tmo, __ATOMIC_RELAXED, __HIP_MEMORY_SCOPE_AGENT) == call || __builtin_amdgcn_s_memrealtime() - t0 > DEADLINE)) { bad = true; break; }
            __builtin_amdgcn_s_sleep(2);
        }
        if (lane == 0) {
            if (bad) {
                __hip_atomic_store(tmo, call, __ATOMIC_RELAXED, __HIP_MEMORY_SCOPE_AGENT);
                *(unsigned short*)(a.out[a.rank & 7] + (size_t)(a.rank * MS + (bid & 31) * 16) * NS * 2 + (bid >> 5) * 128) = 0x7fc0;
            }
            __hip_atomic_store((unsigned*)(myws + CNT_OFF + bid * 64), call, __ATOMIC_RELAXED, __HIP_MEMORY_SCOPE_AGENT);
        }
    }
#undef G
#undef w4
#undef mi_
#undef r8
#undef c8
#undef fr
#undef g4
}
}

extern "C" void kernel_launch(int rank, int nranks, ncclComm_t comm, hipStream_t stream,
                              void* const* d_in, const int* in_sizes, int n_in,
                              void* d_out, int out_size, const int* in_dtypes, int out_dtype,
                              void* const* d_out_all, void* const* d_ws_all, size_t ws_bytes) {
    if (nranks != NR || (unsigned)rank >= (unsigned)NR || ws_bytes < WS_NEED || n_in < 2 || in_sizes[0] != MS * KD || in_sizes[1] != KD * ND || out_size != NR * MS * NS) {
        fprintf(stderr, "kernel_launch: unexpected problem shape\n"); return; }
    Args a = {};
    a.x = (const unsigned short*)d_in[0]; a.w = (const unsigned short*)d_in[1]; a.rank = rank; a.myws = (unsigned char*)d_ws_all[rank];
    for (int p = 0; p < NR; ++p) { a.out[p] = (unsigned char*)d_out_all[p]; a.ws[p] = (unsigned char*)d_ws_all[p]; }
    gemm_a2a<<<NWG, 512, 0, stream>>>(a);
    const hipError_t e = hipGetLastError();
    if (e != hipSuccess) fprintf(stderr, "HIP error %s\n", hipGetErrorString(e));
}
```
